```python
import numpy as np

kernel_name = "gpu_gemm_a2a_m4096_k4096_n16384_e4m3e5m2_dqrelu"


ABSMAX_RTOL = 0.05

COMPUTE_DTYPE = "float32"
EPILOGUE = "dequant_relu"
IN_QUANT = None
OUT_QUANT = None
IN_FP8 = "e4m3e5m2"
IN_DTYPES = {"x": "e4m3", "w_mat": "e5m2",
             "scale_x": "float32", "scale_w": "float32"}
WORLD_SIZE = 8

def _snap_fp8(x, mant_bits, max_abs, min_normal):
    a = np.abs(x.astype(np.float64))
    _m, _e = np.frexp(a)
    q = np.where(a >= min_normal, np.ldexp(1.0, _e - 1 - mant_bits),
                 min_normal * 2.0 ** -mant_bits)
    snapped = np.minimum(np.round(a / q) * q, max_abs)
    return (np.sign(x) * snapped).astype(np.float32)

def setup_inputs():

    x = _snap_fp8(np.random.randn(4096, 4096).astype(np.float32), 3, 448.0, 2.0 ** -6)
    w_mat = _snap_fp8(np.random.randn(4096, 16384).astype(np.float32), 2, 57344.0, 2.0 ** -14)
    scale_x = (0.02 * (1.0 + np.random.rand(1))).astype(np.float32)
    scale_w = (0.02 * (1.0 + np.random.rand(1))).astype(np.float32)
    return {"x": x, "w_mat": w_mat, "scale_x": scale_x, "scale_w": scale_w}

def reference(x, w_mat, scale_x, scale_w):
    acc = (x @ w_mat).astype(np.float32)
    y = (acc * (scale_x * scale_w)).astype(np.float32)
    return np.maximum(y, 0.0).astype(np.float32)
```

```cpp
#include <hip/hip_runtime.h>
#include <rccl/rccl.h>
#include <cstdio>
#include <cstdint>

#define HIP_CHECK(expr) do { hipError_t _e = (expr); if (_e != hipSuccess) { \
    fprintf(stderr, "HIP error %s at %s:%d\n", hipGetErrorString(_e), __FILE__, __LINE__); \
    return; } } while (0)

namespace {

typedef int v8i __attribute__((ext_vector_type(8)));
typedef int v4i __attribute__((ext_vector_type(4)));
typedef int v2i __attribute__((ext_vector_type(2)));
typedef unsigned v4u __attribute__((ext_vector_type(4)));
typedef float v4f __attribute__((ext_vector_type(4)));
#define LDS_AS __attribute__((address_space(3)))

constexpr int NR = 8;
constexpr int MLOC = 512, KDIM = 4096, NTOT = 16384, NLOC = 2048;
constexpr unsigned SHARD = MLOC * KDIM;
constexpr int NRT = MLOC / 16;
constexpr unsigned RT_BYTES = 16 * KDIM;
constexpr size_t STAGE_BYTES = (size_t)2 * NR * SHARD;
constexpr size_t PROG_OFF = STAGE_BYTES;
constexpr size_t CTR_OFF = PROG_OFF + 4096;
constexpr size_t TMO_OFF = CTR_OFF + 4096;
constexpr size_t WS_NEED = TMO_OFF + 128;
constexpr unsigned long long DEADLINE_TICKS = 30000000ull;

constexpr int NPUSH = 28;
constexpr int CONS0 = 32;
constexpr int GRID = 256;

#define PUSH_WIN 3
constexpr int PUSH_PF = 8;

constexpr int LDS_W = 4 * 32768;
constexpr int LDS_RED = 2 * 4 * 3 * 1024;
constexpr int LDS_BYTES = LDS_W + LDS_RED;

struct Params {
  const unsigned char* x;
  const unsigned char* w;
  const float* sx;
  const float* sw;
  float* out;
  unsigned char* ws[NR];
  int rank;
};

template <int N> struct IC { static constexpr int v = N; };
template <int I, int E, class F> __device__ __forceinline__ void static_for(F&& f) {
  if constexpr (I < E) { f(IC<I>{}); static_for<I + 1, E>(f); }
}

__device__ __forceinline__ __amdgpu_buffer_rsrc_t make_rsrc(const void* p, unsigned bytes) {
  const uintptr_t a = (uintptr_t)p;
  const unsigned lo = __builtin_amdgcn_readfirstlane((unsigned)a);
  const unsigned hi = __builtin_amdgcn_readfirstlane((unsigned)(a >> 32));
  void* q = (void*)(((uintptr_t)hi << 32) | lo);
  return __builtin_amdgcn_make_buffer_rsrc(q, 0, bytes, 0x00020000);
}

#define MEMFENCE() asm volatile("" ::: "memory")
template <int N> __device__ __forceinline__ void wait_vm() {
  asm volatile("s_waitcnt vmcnt(%0)" ::"i"(N > 63 ? 63 : N) : "memory");
}

__device__ __forceinline__ void pusher(const Params& P, int b, int wid, int lane, unsigned call) {
  const int pi = b % (NR - 1), u = (b / (NR - 1)) * 4 + wid;
  const unsigned peer = (unsigned)(P.rank + 1 + pi) & (NR - 1);
  if (peer >= (unsigned)NR) return;
  const int w = u >> 2, q = u & 3, m = lane & 15, g = lane >> 4;
  const unsigned slot = call & 1u;
  unsigned char* pws = P.ws[peer];
  const __amdgpu_buffer_rsrc_t rx = make_rsrc(P.x, SHARD);
  const __amdgpu_buffer_rsrc_t rd = make_rsrc(pws + ((size_t)slot * NR + P.rank) * SHARD, SHARD);
  const int vsrc = m * KDIM + g * 16 + w * 1024 + q * 256;
  const int vdst = lane * 16 + w * 16384 + q * 4096;
  unsigned* flag = (unsigned*)(pws + PROG_OFF + (size_t)(P.rank * 4 + w) * 128 + q * 4);
  const unsigned base = (call - 1u) * (unsigned)NRT;

  v4i buf[PUSH_PF][4];
  static_for<0, PUSH_PF>([&](auto C) {
    constexpr int c = decltype(C)::v;
#pragma unroll
    for (int i = 0; i < 4; ++i) buf[c][i] = __builtin_amdgcn_raw_buffer_load_b128(rx, vsrc + i * 64, c * RT_BYTES, 0);
  });
  MEMFENCE();
  static_for<0, NRT + PUSH_WIN - 1>([&](auto C) {
    constexpr int c = decltype(C)::v;
    if constexpr (c < NRT) {
#pragma unroll
      for (int i = 0; i < 4; ++i)
        __builtin_amdgcn_raw_buffer_store_b128(buf[c % PUSH_PF][i], rd, vdst + i * 1024, c * RT_BYTES, 17  );
      MEMFENCE();
      if constexpr (c + PUSH_PF < NRT) {
#pragma unroll
        for (int i = 0; i < 4; ++i)
          buf[c % PUSH_PF][i] = __builtin_amdgcn_raw_buffer_load_b128(rx, vsrc + i * 64, (c + PUSH_PF) * RT_BYTES, 0);
        MEMFENCE();
      }
    }
    constexpr int d = c - PUSH_WIN + 1;
    if constexpr (d >= 0) {
      constexpr auto younger = []() {
        int n = 0;
        for (int j = d; j <= c; ++j) {
          if (j > d && j < NRT) n += 4;
          if (j < NRT && j + PUSH_PF < NRT) n += 4;
          if (j < c && j - PUSH_WIN + 1 >= 0) n += 1;
        }
        return n;
      }();
      wait_vm<younger>();
      if (lane == 0) __hip_atomic_store(flag, base + (unsigned)(d + 1), __ATOMIC_RELAXED, __HIP_MEMORY_SCOPE_SYSTEM);
      MEMFENCE();
    }
  });
}

struct Frags { v4i f[16]; };

__device__ __forceinline__ v4f mfma_wx(const v8i& wf, const v4i& x0, const v4i& x1, const v4f& c) {
  const v8i xf = {x0[0], x0[1], x0[2], x0[3], x1[0], x1[1], x1[2], x1[3]};
  return __builtin_amdgcn_mfma_scale_f32_16x16x128_f8f6f4(wf, xf, c, 1, 0, 0, 0, 0, 0);
}

__device__ __forceinline__ void consumer(const Params& P, int b, int wid, int lane, unsigned call,
                                         LDS_AS unsigned char* lds) {
  const int c = b - CONS0;
  const int xcd = c & 7, idx = c >> 3;
  const int j = xcd * 4 + (idx & 3);
  const int pi = idx >> 2;
  const unsigned src = (unsigned)(P.rank + 1 + pi) & (NR - 1);
  const int m = lane & 15, g = lane >> 4;
  const unsigned slot = call & 1u;
  unsigned char* ws = P.ws[P.rank];
  const int n_own = pi < 2 ? 6 : 4;
  const int own0 = pi < 2 ? 6 * pi : 12 + 4 * (pi - 2);
  const int NB = n_own + NRT;

  const uintptr_t xaddr = (uintptr_t)P.x;
  const uintptr_t saddr = (uintptr_t)(ws + ((size_t)slot * NR + src) * SHARD);
  const int voff_own = m * KDIM + g * 16 + wid * 1024;
  const int voff_rem = lane * 16 + wid * 16384;
  const unsigned* prog = (const unsigned*)(ws + PROG_OFF + (size_t)(src * 4 + wid) * 128) + (lane & 3);
  unsigned* tmo = (unsigned*)(ws + TMO_OFF);
  const unsigned base = (call - 1u) * (unsigned)NRT;
  const float scale = P.sx[0] * P.sw[0];

  struct LCtx { __amdgpu_buffer_rsrc_t r; int voff, soff, stride; };
  auto make_ctx = [&](int t, bool ok) -> LCtx {
    const bool own = t < n_own;
    const uintptr_t a = own ? xaddr : saddr;
    LCtx c;
    c.r = __builtin_amdgcn_make_buffer_rsrc((void*)a, 0, ok ? SHARD : 0u, 0x00020000);
    c.voff = own ? voff_own : voff_rem;
    c.soff = (own ? own0 + t : t - n_own) * (int)RT_BYTES;
    c.stride = own ? 64 : 1024;
    return c;
  };
  auto load_frag = [&](const LCtx& c, int i) -> v4i {
    return __builtin_amdgcn_raw_buffer_load_b128(c.r, c.voff, c.soff + i * c.stride, 0);
  };

  Frags A0, A1;
  {
    const LCtx c0 = make_ctx(0, true), c1 = make_ctx(1, true);
#pragma unroll
    for (int i = 0; i < 16; ++i) A0.f[i] = load_frag(c0, i);
#pragma unroll
    for (int i = 0; i < 16; ++i) A1.f[i] = load_frag(c1, i);
  }

  v8i B[8][4];
  {
    LDS_AS unsigned char* wl = lds + wid * 32768;
    const unsigned char* wg = P.w + (size_t)(wid * 1024) * NTOT + (size_t)P.rank * NLOC + j * 64;
    const int r0 = lane >> 2, cp = lane & 3;
    auto stage = [&](int half, int ks0) {
#pragma unroll
      for (int n = 0; n < 16; ++n) {
        const int r = n * 16 + r0;
        const int f = ((r >> 2) & 1) | (((r >> 4) & 1) << 1);
        const unsigned char* gp = wg + (size_t)(ks0 * 128 + r) * NTOT + ((cp ^ f) << 4);
        __builtin_amdgcn_global_load_lds((const unsigned*)gp, (LDS_AS unsigned*)(wl + half * 16384 + n * 1024), 16, 0, 0);
      }
    };
    const int q = (lane & 15) >> 1, p = lane & 1;
    auto fetch = [&](int half, auto KS0) {
      constexpr int ks0 = decltype(KS0)::v;
#pragma unroll
      for (int kk = 0; kk < 2; ++kk)
#pragma unroll
        for (int cl = 0; cl < 4; ++cl) {
          const int ct = (cl + wid) & 3;
          v2i t[4];
#pragma unroll
          for (int i = 0; i < 4; ++i) {
            const int r = kk * 128 + (i >> 1) * 64 + g * 16 + (i & 1) * 8 + q;
            const int f = ((r >> 2) & 1) | (((r >> 4) & 1) << 1);
            t[i] = __builtin_amdgcn_ds_read_tr8_b64_v2i32((LDS_AS v2i*)(wl + half * 16384 + r * 64 + ((ct ^ f) << 4) + p * 8));
          }
          B[ks0 + kk][cl] = v8i{t[0][0], t[0][1], t[1][0], t[1][1], t[2][0], t[2][1], t[3][0], t[3][1]};
        }
    };
    stage(0, 0); stage(1, 2);
    wait_vm<16>(); fetch(0, IC<0>{});
    asm volatile("s_waitcnt lgkmcnt(0)" ::: "memory");
    stage(0, 4);
    wait_vm<16>(); fetch(1, IC<2>{});
    asm volatile("s_waitcnt lgkmcnt(0)" ::: "memory");
    stage(1, 6);
    wait_vm<16>(); fetch(0, IC<4>{});
    wait_vm<0>(); fetch(1, IC<6>{});
    asm volatile("s_waitcnt lgkmcnt(0)" ::: "memory");
  }

  LDS_AS unsigned char* red = lds + LDS_W;
  float* outp = P.out + (size_t)m * NLOC + j * 64 + wid * 16 + g * 4;
  int avail = n_own;
  bool dead = false;
  bool valid0 = true, valid1 = true;
  v4f keep = {0.f, 0.f, 0.f, 0.f};

  auto read_prog = [&]() -> int {
    const unsigned v = __hip_atomic_load(prog, __ATOMIC_RELAXED, __HIP_MEMORY_SCOPE_SYSTEM) - base;
    int a = (int)v;
    a = a < 0 ? 0 : a;
    const int a0 = __builtin_amdgcn_readlane(a, 0), a1 = __builtin_amdgcn_readlane(a, 1);
    const int a2 = __builtin_amdgcn_readlane(a, 2), a3 = __builtin_amdgcn_readlane(a, 3);
    const int mn = min(min(a0, a1), min(a2, a3));
    return n_own + min(mn, NRT);
  };
  const int row_own = P.rank * MLOC + own0 * 16, row_rem = (int)src * MLOC - n_own * 16;
  auto out_row = [&](int t) -> size_t { return (size_t)(unsigned)((t < n_own ? row_own : row_rem) + t * 16); };
  auto finish = [&](int t, const v4f& p0, const v4f& p1, const v4f& p2) {
    v4f y = ((keep + p0) + (p1 + p2)) * scale;
#pragma unroll
    for (int e = 0; e < 4; ++e) y[e] = y[e] < 0.f ? 0.f : y[e];
    *(v4f*)(outp + out_row(t) * NLOC) = y;
  };

  auto step = [&](int t, Frags& A, bool& valid, Frags& Ao, bool& valido) {
    if (!valid) {
      if (avail <= t && !dead) {
        const unsigned long long t0 = __builtin_amdgcn_s_memrealtime();
        for (unsigned polls = 1;; ++polls) {
          avail = read_prog();
          if (avail > t) break;
          if ((polls & 63u) == 0u &&
              (__builtin_amdgcn_readfirstlane(__hip_atomic_load(tmo, __ATOMIC_RELAXED, __HIP_MEMORY_SCOPE_AGENT)) == call ||
               __builtin_amdgcn_s_memrealtime() - t0 > DEADLINE_TICKS)) {
            __hip_atomic_store(tmo, call, __ATOMIC_RELAXED, __HIP_MEMORY_SCOPE_AGENT);
            dead = true;
            break;
          }
          __builtin_amdgcn_s_sleep(1);
        }
      }
      const LCtx c0 = make_ctx(t, !dead);
#pragma unroll
      for (int i = 0; i < 16; ++i) A.f[i] = load_frag(c0, i);
      if (!valido && avail > t + 1) {
        const LCtx c1 = make_ctx(t + 1, true);
#pragma unroll
        for (int i = 0; i < 16; ++i) Ao.f[i] = load_frag(c1, i);
        valido = true;
      }
    }
    unsigned pv = 0;
    const bool want_poll = avail < NB && !dead;
    if (want_poll) pv = __hip_atomic_load(prog, __ATOMIC_RELAXED, __HIP_MEMORY_SCOPE_SYSTEM);
    const bool nxt = avail > t + 2;
    const LCtx cn = make_ctx(t + 2, nxt);
    v4f acc[4];
    const float z = dead ? __builtin_nanf("") : 0.f;
#pragma unroll
    for (int cl = 0; cl < 4; ++cl) acc[cl] = v4f{z, z, z, z};
    v4f p0, p1, p2;
    LDS_AS unsigned char* rprev = red + ((t + 1) & 1) * 12288 + wid * 3072 + lane * 16;
#pragma unroll
    for (int ks = 0; ks < 8; ++ks) {
#pragma unroll
      for (int cl = 0; cl < 4; ++cl) acc[cl] = mfma_wx(B[ks][cl], A.f[2 * ks], A.f[2 * ks + 1], acc[cl]);
      A.f[2 * ks] = load_frag(cn, 2 * ks);
      A.f[2 * ks + 1] = load_frag(cn, 2 * ks + 1);
      if (ks == 2 && t > 0) {
        asm volatile("s_waitcnt lgkmcnt(0)" ::: "memory");
        __builtin_amdgcn_s_barrier();
        MEMFENCE();
        p0 = *(LDS_AS v4f*)(rprev);
        p1 = *(LDS_AS v4f*)(rprev + 1024);
        p2 = *(LDS_AS v4f*)(rprev + 2048);
      }
      if (ks == 5 && t > 0) finish(t - 1, p0, p1, p2);
      __builtin_amdgcn_sched_barrier(0);
    }
    valid = nxt;
    LDS_AS unsigned char* rcur = red + (t & 1) * 12288 + lane * 16;
#pragma unroll
    for (int cl = 1; cl < 4; ++cl) {
      const int d = (cl + wid) & 3;
      *(LDS_AS v4f*)(rcur + d * 3072 + (cl - 1) * 1024) = acc[cl];
    }
    keep = acc[0];
    if (want_poll) {
      int a = (int)(pv - base);
      a = a < 0 ? 0 : a;
      const int a0 = __builtin_amdgcn_readlane(a, 0), a1 = __builtin_amdgcn_readlane(a, 1);
      const int a2 = __builtin_amdgcn_readlane(a, 2), a3 = __builtin_amdgcn_readlane(a, 3);
      avail = n_own + min(min(min(a0, a1), min(a2, a3)), NRT);
    }
  };

  for (int t = 0; t < NB; t += 2) {
    step(t, A0, valid0, A1, valid1);
    step(t + 1, A1, valid1, A0, valid0);
  }
  {
    asm volatile("s_waitcnt lgkmcnt(0)" ::: "memory");
    __builtin_amdgcn_s_barrier();
    MEMFENCE();
    LDS_AS unsigned char* rprev = red + ((NB - 1) & 1) * 12288 + wid * 3072 + lane * 16;
    const v4f p0 = *(LDS_AS v4f*)(rprev), p1 = *(LDS_AS v4f*)(rprev + 1024), p2 = *(LDS_AS v4f*)(rprev + 2048);
    finish(NB - 1, p0, p1, p2);
  }
}

__global__ void __launch_bounds__(256, 1) gemm_ag_kernel(const Params P) {
  __shared__ __attribute__((aligned(16))) unsigned char lds[LDS_BYTES];
  const int b = blockIdx.x;
  const int wid = __builtin_amdgcn_readfirstlane(threadIdx.x >> 6), lane = threadIdx.x & 63;
  if (b >= NPUSH && b < CONS0) return;
  unsigned* ctr = (unsigned*)(P.ws[P.rank] + CTR_OFF) + b * 4 + wid;
  unsigned tk = 0;
  if (lane == 0) tk = __hip_atomic_fetch_add(ctr, 1u, __ATOMIC_RELAXED, __HIP_MEMORY_SCOPE_AGENT);
  const unsigned call = __builtin_amdgcn_readfirstlane(tk) + 1u;
  if (b < NPUSH) pusher(P, b, wid, lane, call);
  else consumer(P, b, wid, lane, call, (LDS_AS unsigned char*)lds);
}

}

extern "C" void kernel_launch(int rank, int nranks, ncclComm_t comm,
                              hipStream_t stream,
                              void* const* d_in, const int* in_sizes, int n_in,
                              void* d_out, int out_size,
                              const int* in_dtypes, int out_dtype,
                              void* const* d_out_all,
                              void* const* d_ws_all, size_t ws_bytes) {
  (void)comm; (void)in_dtypes; (void)out_dtype; (void)d_out_all;
  if (nranks != NR || n_in < 4 || ws_bytes < WS_NEED || in_sizes[0] != MLOC * KDIM || in_sizes[1] != KDIM * NTOT ||
      out_size != NR * MLOC * NLOC || (unsigned)rank >= (unsigned)NR) {
    fprintf(stderr, "kernel_launch: unexpected problem shape\n");
    return;
  }
  Params P;
  P.x = (const unsigned char*)d_in[0];
  P.w = (const unsigned char*)d_in[1];
  P.sx = (const float*)d_in[2];
  P.sw = (const float*)d_in[3];
  P.out = (float*)d_out;
  for (int p = 0; p < NR; ++p) P.ws[p] = (unsigned char*)d_ws_all[p];
  P.rank = rank;
  hipLaunchKernelGGL(gemm_ag_kernel, dim3(GRID), dim3(256), 0, stream, P);
  HIP_CHECK(hipGetLastError());
}
```

```cpp
#include <hip/hip_runtime.h>
#include <rccl/rccl.h>
#include <cstdio>
#include <cstdint>

#define HIP_CHECK(expr) do { hipError_t _e = (expr); if (_e != hipSuccess) { \
    fprintf(stderr, "HIP error %s at %s:%d\n", hipGetErrorString(_e), __FILE__, __LINE__); \
    return; } } while (0)

namespace {

typedef int v4i __attribute__((ext_vector_type(4)));
#define LDS_AS __attribute__((address_space(3)))

constexpr int NR = 8;
constexpr int MLOC = 512, KDIM = 4096, NTOT = 16384, NLOC = 2048;
constexpr unsigned SHARD = MLOC * KDIM;
constexpr int NRT = MLOC / 16;
constexpr unsigned RT_BYTES = 16 * KDIM;
constexpr size_t STAGE_BYTES = (size_t)2 * NR * SHARD;
constexpr size_t PROG_OFF = STAGE_BYTES;
constexpr size_t CTR_OFF = PROG_OFF + 4096;
constexpr size_t TMO_OFF = CTR_OFF + 4096;
constexpr size_t CLR_OFF = TMO_OFF + 128;
constexpr size_t WS_NEED = CLR_OFF + 128;
constexpr int ZONE_K = 8;
constexpr int RFIRST = 4;
constexpr unsigned long long DEADLINE_TICKS = 30000000ull;

#define PUSH_PF 4
#define FLAG_EVERY 2
constexpr int NPUSH = 16;
constexpr int NCLR = 16;
constexpr int CONS0 = 32;
constexpr int GRID = 256;

constexpr int LDS_RED = 32768;
constexpr int LDS_BYTES = LDS_RED + 3 * 12288 + 4096;

struct Params {
  const unsigned char* x;
  unsigned char* ws;
  int rank, pad;
  unsigned char* peer[NR - 1];
  const unsigned char* w;
  const float* sx;
  const float* sw;
  float* out;
};

__device__ __forceinline__ __amdgpu_buffer_rsrc_t make_rsrc(const void* p, unsigned bytes) {
  const uintptr_t a = (uintptr_t)p;
  const unsigned lo = __builtin_amdgcn_readfirstlane((unsigned)a);
  const unsigned hi = __builtin_amdgcn_readfirstlane((unsigned)(a >> 32));
  void* q = (void*)(((uintptr_t)hi << 32) | lo);
  return __builtin_amdgcn_make_buffer_rsrc(q, 0, bytes, 0x00020000);
}

#define MEMFENCE() asm volatile("" ::: "memory")
template <int N> __device__ __forceinline__ void wait_vm() {
  asm volatile("s_waitcnt vmcnt(%0)" ::"i"(N > 63 ? 63 : N) : "memory");
}
template <int N> struct IC { static constexpr int v = N; };
template <int I, int E, class F> __device__ __forceinline__ void static_for(F&& f) {
  if constexpr (I < E) { f(IC<I>{}); static_for<I + 1, E>(f); }
}

__device__ __forceinline__ void pusher(const Params& P, int qd, int wid, int lane, unsigned ticket, unsigned* ctr,
                                       volatile LDS_AS unsigned* failw, unsigned tstart) {
  constexpr int NPP = NR - 1;
  const int pc = qd * 4 + wid;
  const int m = lane & 15, g = lane >> 4;
  const __amdgpu_buffer_rsrc_t rx = make_rsrc(P.x, SHARD);
  const int vsrc = m * KDIM + g * 16 + pc * 64;
  const int vdst = lane * 16 + pc * 1024;
  v4i buf[NRT];
#pragma unroll
  for (int c = 0; c < PUSH_PF; ++c) buf[c] = __builtin_amdgcn_raw_buffer_load_b128(rx, vsrc, c * RT_BYTES, 0);
  MEMFENCE();
  const unsigned call = __builtin_amdgcn_readfirstlane(ticket) + 1u;
  const unsigned slot = call & 1u;
  const size_t soff = ((size_t)slot * NR + P.rank) * SHARD;
  __amdgpu_buffer_rsrc_t rd[NPP];
#pragma unroll
  for (int p = 0; p < NPP; ++p) rd[p] = make_rsrc(P.peer[p] + soff, SHARD);
  auto st = [&](auto C) {
    constexpr int c = decltype(C)::v;
#pragma unroll
    for (int p = 0; p < NPP; ++p) __builtin_amdgcn_raw_buffer_store_b128(buf[c], rd[p], vdst, c * RT_BYTES, 17  );
  };
  st(IC<0>{});
  MEMFENCE();
  if (lane == 0) __hip_atomic_store(ctr, call, __ATOMIC_RELAXED, __HIP_MEMORY_SCOPE_AGENT);
  unsigned flo = 0, fhi = 0;
#pragma unroll
  for (int p = 0; p < NPP; ++p) {
    const unsigned long long a = (unsigned long long)(uintptr_t)P.peer[p];
    flo = lane == p ? (unsigned)a : flo;
    fhi = lane == p ? (unsigned)(a >> 32) : fhi;
  }
  typedef __attribute__((address_space(1))) unsigned gu32;
  gu32* flag = (gu32*)((((unsigned long long)fhi << 32) | flo) + PROG_OFF + (size_t)(P.rank * 4 + (qd >> 2)) * 128 + (qd & 3) * 4);
  const unsigned base = (call - 1u) * (unsigned)NRT;
  const unsigned* clrc = (const unsigned*)(P.ws + CLR_OFF);
  unsigned clr = 0;
  int clr_left = 1;
  bool gate_ok = true;
  if (wid == 0 && lane == 0) *failw = 0u;
  MEMFENCE();
  static_for<0, NRT>([&](auto C) {
    constexpr int c = decltype(C)::v;
    if constexpr (c + PUSH_PF < NRT) buf[c + PUSH_PF] = __builtin_amdgcn_raw_buffer_load_b128(rx, vsrc, (c + PUSH_PF) * RT_BYTES, 0);
    MEMFENCE();
    if constexpr (c + 1 < NRT) {
      if constexpr (ZONE_K > 0 && c + 2 == NRT) {
        if (clr_left > 0) {
          const unsigned long long t0 = __builtin_amdgcn_s_memrealtime();
#pragma nounroll
          while (clr_left > 0) {
            if (__builtin_amdgcn_s_memrealtime() - t0 > DEADLINE_TICKS) { gate_ok = false; *failw = 1u; break; }
            clr_left = (int)(call * (unsigned)NCLR - __builtin_amdgcn_readfirstlane(__hip_atomic_load(clrc, __ATOMIC_RELAXED, __HIP_MEMORY_SCOPE_AGENT)));
          }
        }
      }
      if constexpr (ZONE_K > 0 && c + 4 == NRT) { clr = __hip_atomic_load(clrc, __ATOMIC_RELAXED, __HIP_MEMORY_SCOPE_AGENT); MEMFENCE(); }
      if (gate_ok) st(IC<c + 1>{});
    }
    MEMFENCE();
    wait_vm<(c + 1 < NRT ? NPP : 0)>();
    if constexpr (ZONE_K > 0 && c + 4 == NRT) clr_left = (int)(call * (unsigned)NCLR - __builtin_amdgcn_readfirstlane(clr));
    if constexpr (c == 0 || c + 2 == NRT) asm volatile("s_waitcnt lgkmcnt(0)" ::: "memory");
    __builtin_amdgcn_s_barrier();
    MEMFENCE();
    if constexpr (c < NRT - ZONE_K || (NRT - 1 - c) % FLAG_EVERY == 0) {
      if (wid == (c & 3) && lane < NPP && (c + 1 < NRT || *failw == 0u))
        __hip_atomic_store(flag, base + (unsigned)(c + 1), __ATOMIC_RELAXED, __HIP_MEMORY_SCOPE_SYSTEM);
    }
    MEMFENCE();
  });
}

__device__ __forceinline__ void helper(const Params& P, int qd, int wid, unsigned* ctr) {
  const int lane = __builtin_amdgcn_mbcnt_hi(~0u, __builtin_amdgcn_mbcnt_lo(~0u, 0u));
  const unsigned tk = __hip_atomic_load(ctr, __ATOMIC_RELAXED, __HIP_MEMORY_SCOPE_AGENT);
  const int pc = qd * 4 + wid, m = lane & 15, g = lane >> 4;
  const __amdgpu_buffer_rsrc_t rx = make_rsrc(P.x, SHARD);
  const int vsrc = m * KDIM + g * 16 + pc * 64, vdst = lane * 16 + pc * 1024;
  constexpr int HB = 8;
  v4i buf[2][HB];
#pragma unroll
  for (int k = 0; k < HB; ++k) buf[0][k] = __builtin_amdgcn_raw_buffer_load_b128(rx, vsrc, k * RT_BYTES, 0);
  MEMFENCE();
  const unsigned call = __builtin_amdgcn_readfirstlane(tk) + 1u;
  if (lane == 0) __hip_atomic_store(ctr, call, __ATOMIC_RELAXED, __HIP_MEMORY_SCOPE_AGENT);
  const __amdgpu_buffer_rsrc_t ro = make_rsrc(P.ws + ((size_t)(call & 1u) * NR + P.rank) * SHARD, SHARD);
  unsigned* flag = (unsigned*)(P.ws + PROG_OFF + (size_t)(P.rank * 4 + (qd >> 2)) * 128 + (qd & 3) * 4);
  static_for<0, NRT / HB>([&](auto B) {
    constexpr int bb = decltype(B)::v;
    if constexpr ((bb + 1) * HB < NRT) {
#pragma unroll
      for (int k = 0; k < HB; ++k) buf[(bb + 1) & 1][k] = __builtin_amdgcn_raw_buffer_load_b128(rx, vsrc, ((bb + 1) * HB + k) * RT_BYTES, 0);
    }
    MEMFENCE();
#pragma unroll
    for (int k = 0; k < HB; ++k) __builtin_amdgcn_raw_buffer_store_b128(buf[bb & 1][k], ro, vdst, (bb * HB + k) * RT_BYTES, 17  );
    MEMFENCE();
  });
  asm volatile("s_waitcnt vmcnt(0)" ::: "memory");
  __builtin_amdgcn_s_barrier();
  if (wid == 0 && lane == 0) __hip_atomic_store(flag, call, __ATOMIC_RELAXED, __HIP_MEMORY_SCOPE_AGENT);
  MEMFENCE();
  if constexpr (ZONE_K > 0) {
    const __amdgpu_buffer_rsrc_t rs = make_rsrc(P.ws + (size_t)((call + 1u) & 1u) * NR * SHARD, NR * SHARD);
    const v4i z = {0, 0, 0, 0};
    for (int p = 0; p < NR - 1; ++p) {
      const unsigned src = (unsigned)(P.rank + 1 + p) & (NR - 1);
#pragma unroll
      for (int k = 0; k < ZONE_K; ++k)
        __builtin_amdgcn_raw_buffer_store_b128(z, rs, vdst, src * SHARD + (NRT - ZONE_K + k) * RT_BYTES, 17  );
    }
    asm volatile("s_waitcnt vmcnt(0)" ::: "memory");
    __builtin_amdgcn_s_barrier();
    if (wid == 0 && lane == 0) (void)__hip_atomic_fetch_add((unsigned*)(P.ws + CLR_OFF), 1u, __ATOMIC_RELAXED, __HIP_MEMORY_SCOPE_AGENT);
  }
}

__device__ __forceinline__ void consumer(const Params& P, int b, int wid, unsigned lds_base) {
  const int c = b - CONS0;
  const int xcd = c & 7, idx = c >> 3;
  const int j = xcd * 4 + (idx & 3);
  const int pi = idx >> 2;
  const int rot = (pi + 2 * (idx & 3)) & 7;
  const unsigned src = (unsigned)(P.rank + 1 + pi) & (NR - 1);
  unsigned char* ws = P.ws;
  const int n_own = 4 + (pi < 4);
  const int nb = n_own + NRT;
  const unsigned char* waddr = P.w + (size_t)(wid * 1024) * NTOT + j * 64;
  static_assert(PROG_OFF == 32u << 20 && TMO_OFF - PROG_OFF == 8192, "offsets are baked into the asm");
  const int ocol = j * 256 + wid * 64;
  asm volatile(
      "s_nop 4\n"
      "v_mbcnt_lo_u32_b32 v16, -1, 0\n"
      "v_mbcnt_hi_u32_b32 v16, -1, v16\n"
      "v_lshrrev_b32 v17, 2, v16\n"
      "v_and_b32 v18, 3, v16\n"
      "v_lshlrev_b32 v5, 14, v17\n"
      "v_lshl_add_u32 v5, v18, 4, v5\n"
      "s_mov_b64 s[98:99], %[fl]\n"
      "s_mov_b64 s[36:37], %[fl]\n"
      "s_and_b32 s37, s37, 0xffff\n"
      "s_mov_b32 s39, 0x00020000\n"
      "v_lshlrev_b32 v4, 2, v18\n"
      "s_mov_b64 s[64:65], %[waddr]\n"
      "s_and_b32 s65, s65, 0xffff\n"
      "s_mov_b32 s66, 16760896\n"
      "s_mov_b32 s67, 0x00020000\n"
      "v_mov_b32 v9, 0\n"
      "v_mov_b32 v28, %[cto]\n"
      "global_load_dword v8, v28, s[98:99] sc1\n"
      "global_load_dword v10, v9, %[sxp]\n"
      "global_load_dword v25, v9, %[swp]\n"
      "s_lshl_b32 s85, %[rot], 21\n"
      "buffer_load_dwordx4 a[0:3], v5, s[64:67], s85 offen\n"
      "s_add_u32 s85, s85, 262144\n"
      "buffer_load_dwordx4 a[4:7], v5, s[64:67], s85 offen\n"
      "s_add_u32 s85, s85, 262144\n"
      "buffer_load_dwordx4 a[8:11], v5, s[64:67], s85 offen\n"
      "s_add_u32 s85, s85, 262144\n"
      "buffer_load_dwordx4 a[12:15], v5, s[64:67], s85 offen\n"
      "s_add_u32 s85, s85, 262144\n"
      "buffer_load_dwordx4 a[16:19], v5, s[64:67], s85 offen\n"
      "s_add_u32 s85, s85, 262144\n"
      "buffer_load_dwordx4 a[20:23], v5, s[64:67], s85 offen\n"
      "s_add_u32 s85, s85, 262144\n"
      "buffer_load_dwordx4 a[24:27], v5, s[64:67], s85 offen\n"
      "s_add_u32 s85, s85, 262144\n"
      "buffer_load_dwordx4 a[28:31], v5, s[64:67], s85 offen\n"
      "s_add_u32 s85, s85, 262144\n"
      "s_and_b32 s85, s85, 0xffffff\n"
      "buffer_load_dwordx4 a[32:35], v5, s[64:67], s85 offen\n"
      "s_add_u32 s85, s85, 262144\n"
      "buffer_load_dwordx4 a[36:39], v5, s[64:67], s85 offen\n"
      "s_add_u32 s85, s85, 262144\n"
      "buffer_load_dwordx4 a[40:43], v5, s[64:67], s85 offen\n"
      "s_add_u32 s85, s85, 262144\n"
      "buffer_load_dwordx4 a[44:47], v5, s[64:67], s85 offen\n"
      "s_add_u32 s85, s85, 262144\n"
      "buffer_load_dwordx4 a[48:51], v5, s[64:67], s85 offen\n"
      "s_add_u32 s85, s85, 262144\n"
      "buffer_load_dwordx4 a[52:55], v5, s[64:67], s85 offen\n"
      "s_add_u32 s85, s85, 262144\n"
      "buffer_load_dwordx4 a[56:59], v5, s[64:67], s85 offen\n"
      "s_add_u32 s85, s85, 262144\n"
      "buffer_load_dwordx4 a[60:63], v5, s[64:67], s85 offen\n"
      "s_add_u32 s85, s85, 262144\n"
      "s_and_b32 s85, s85, 0xffffff\n"
      "buffer_load_dwordx4 a[64:67], v5, s[64:67], s85 offen\n"
      "s_add_u32 s85, s85, 262144\n"
      "buffer_load_dwordx4 a[68:71], v5, s[64:67], s85 offen\n"
      "s_add_u32 s85, s85, 262144\n"
      "buffer_load_dwordx4 a[72:75], v5, s[64:67], s85 offen\n"
      "s_add_u32 s85, s85, 262144\n"
      "buffer_load_dwordx4 a[76:79], v5, s[64:67], s85 offen\n"
      "s_add_u32 s85, s85, 262144\n"
      "buffer_load_dwordx4 a[80:83], v5, s[64:67], s85 offen\n"
      "s_add_u32 s85, s85, 262144\n"
      "buffer_load_dwordx4 a[84:87], v5, s[64:67], s85 offen\n"
      "s_add_u32 s85, s85, 262144\n"
      "buffer_load_dwordx4 a[88:91], v5, s[64:67], s85 offen\n"
      "s_add_u32 s85, s85, 262144\n"
      "buffer_load_dwordx4 a[92:95], v5, s[64:67], s85 offen\n"
      "s_add_u32 s85, s85, 262144\n"
      "s_and_b32 s85, s85, 0xffffff\n"
      "s_lshl_b32 s94, %[wid], 13\n"
      "s_add_u32 s94, s94, %[ldsb]\n"
      "v_bfe_u32 v19, v16, 4, 1\n"
      "v_xor_b32 v19, v19, v18\n"
      "v_lshlrev_b32 v19, 4, v19\n"
      "v_lshl_add_u32 v6, v17, 6, v19\n"
      "v_add_u32 v6, s94, v6\n"
      "v_and_b32 v20, 15, v16\n"
      "v_and_b32 v21, 48, v16\n"
      "v_bfe_u32 v22, v16, 1, 3\n"
      "v_or_b32 v22, v22, v21\n"
      "v_lshlrev_b32 v22, 6, v22\n"
      "v_bfe_u32 v23, v16, 3, 2\n"
      "v_xor_b32 v23, %[wid], v23\n"
      "v_lshl_add_u32 v22, v23, 4, v22\n"
      "v_and_b32 v23, 1, v16\n"
      "v_lshl_add_u32 v22, v23, 3, v22\n"
      "v_add_u32 v7, s94, v22\n"
      "s_lshl_b32 s95, %[wid], 10\n"
      "v_lshlrev_b32 v24, 4, v16\n"
      "s_lshl_b32 s96, %[wid], 14\n"
      "v_add_u32 v1, s96, v24\n"
      "v_lshl_add_u32 v2, v20, 13, v21\n"
      "v_add_u32 v2, %[ocol], v2\n"
      "s_add_u32 s95, s95, %[ldsb]\n"
      "s_add_u32 s95, s95, 32768\n"
      "v_add_u32 v3, s95, v24\n"
      "s_mov_b64 s[68:69], %[out]\n"
      "s_and_b32 s69, s69, 0xffff\n"
      "s_mov_b32 s70, 0\n"
      "s_mov_b32 s71, 0x00020000\n"
      "s_mov_b32 s72, 0\n"
      "s_mov_b32 s73, 0\n"
      "s_mov_b32 s74, 0\n"
      "s_mov_b32 s46, 0\n"
      "s_sub_u32 s47, %[nb], 8\n"
      "s_mov_b32 s94, 0\n"
      "s_mul_i32 s96, s94, 7\n"
      "s_add_u32 s96, s96, %[ownu]\n"
      "s_add_u32 s95, s94, %[srcu]\n"
      "s_cmp_lt_u32 s94, %[nownr]\n"
      "s_cselect_b32 s43, s96, s95\n"
      "s_add_u32 s95, s95, %[nown]\n"
      "s_cmp_lt_u32 s94, 4\n"
      "s_cselect_b32 s43, s95, s43\n"
      "s_mov_b32 s94, 1\n"
      "s_mul_i32 s96, s94, 7\n"
      "s_add_u32 s96, s96, %[ownu]\n"
      "s_add_u32 s95, s94, %[srcu]\n"
      "s_cmp_lt_u32 s94, %[nownr]\n"
      "s_cselect_b32 s44, s96, s95\n"
      "s_add_u32 s95, s95, %[nown]\n"
      "s_cmp_lt_u32 s94, 4\n"
      "s_cselect_b32 s44, s95, s44\n"
      "s_mov_b32 s42, 0\n"
      "s_lshl_b32 s48, %[rot], 11\n"
      "s_add_u32 s49, s48, 1024\n"
      "s_add_u32 s50, s49, 1024\n"
      "s_and_b32 s50, s50, 0x3fff\n"
      "s_add_u32 s51, s50, 1024\n"
      "s_add_u32 s52, s51, 1024\n"
      "s_and_b32 s52, s52, 0x3fff\n"
      "s_add_u32 s53, s52, 1024\n"
      "s_add_u32 s54, s53, 1024\n"
      "s_and_b32 s54, s54, 0x3fff\n"
      "s_add_u32 s55, s54, 1024\n"
      "s_add_u32 s56, s55, 1024\n"
      "s_and_b32 s56, s56, 0x3fff\n"
      "s_add_u32 s57, s56, 1024\n"
      "s_add_u32 s58, s57, 1024\n"
      "s_and_b32 s58, s58, 0x3fff\n"
      "s_add_u32 s59, s58, 1024\n"
      "s_add_u32 s60, s59, 1024\n"
      "s_and_b32 s60, s60, 0x3fff\n"
      "s_add_u32 s61, s60, 1024\n"
      "s_add_u32 s62, s61, 1024\n"
      "s_and_b32 s62, s62, 0x3fff\n"
      "s_add_u32 s63, s62, 1024\n"
      "v_xor_b32 v16, 32, v6\n"
      "v_xor_b32 v17, 16, v7\n"
      "v_xor_b32 v18, 32, v7\n"
      "v_xor_b32 v19, 48, v7\n"
      "s_waitcnt vmcnt(16)\n"
      "buffer_load_dwordx4 a[96:99], v5, s[64:67], s85 offen\n"
      "s_add_u32 s85, s85, 262144\n"
      "buffer_load_dwordx4 a[100:103], v5, s[64:67], s85 offen\n"
      "s_add_u32 s85, s85, 262144\n"
      "buffer_load_dwordx4 a[104:107], v5, s[64:67], s85 offen\n"
      "s_add_u32 s85, s85, 262144\n"
      "buffer_load_dwordx4 a[108:111], v5, s[64:67], s85 offen\n"
      "s_add_u32 s85, s85, 262144\n"
      "buffer_load_dwordx4 a[112:115], v5, s[64:67], s85 offen\n"
      "s_add_u32 s85, s85, 262144\n"
      "buffer_load_dwordx4 a[116:119], v5, s[64:67], s85 offen\n"
      "s_add_u32 s85, s85, 262144\n"
      "buffer_load_dwordx4 a[120:123], v5, s[64:67], s85 offen\n"
      "s_add_u32 s85, s85, 262144\n"
      "buffer_load_dwordx4 a[124:127], v5, s[64:67], s85 offen\n"
      "s_add_u32 s85, s85, 262144\n"
      "s_and_b32 s85, s85, 0xffffff\n"
      "ds_write_b128 v6, a[0:3] offset:0\n"
      "ds_write_b128 v16, a[4:7] offset:1024\n"
      "ds_write_b128 v6, a[8:11] offset:2048\n"
      "ds_write_b128 v16, a[12:15] offset:3072\n"
      "ds_write_b128 v6, a[16:19] offset:4096\n"
      "ds_write_b128 v16, a[20:23] offset:5120\n"
      "ds_write_b128 v6, a[24:27] offset:6144\n"
      "ds_write_b128 v16, a[28:31] offset:7168\n"
      "ds_read_b64_tr_b8 a[0:1], v7 offset:0\n"
      "ds_read_b64_tr_b8 a[2:3], v7 offset:512\n"
      "ds_read_b64_tr_b8 a[4:5], v7 offset:4096\n"
      "ds_read_b64_tr_b8 a[6:7], v7 offset:4608\n"
      "ds_read_b64_tr_b8 a[8:9], v17 offset:0\n"
      "ds_read_b64_tr_b8 a[10:11], v17 offset:512\n"
      "ds_read_b64_tr_b8 a[12:13], v17 offset:4096\n"
      "ds_read_b64_tr_b8 a[14:15], v17 offset:4608\n"
      "ds_read_b64_tr_b8 a[16:17], v18 offset:0\n"
      "ds_read_b64_tr_b8 a[18:19], v18 offset:512\n"
      "ds_read_b64_tr_b8 a[20:21], v18 offset:4096\n"
      "ds_read_b64_tr_b8 a[22:23], v18 offset:4608\n"
      "ds_read_b64_tr_b8 a[24:25], v19 offset:0\n"
      "ds_read_b64_tr_b8 a[26:27], v19 offset:512\n"
      "ds_read_b64_tr_b8 a[28:29], v19 offset:4096\n"
      "ds_read_b64_tr_b8 a[30:31], v19 offset:4608\n"
      "v_readfirstlane_b32 s94, v8\n"
      "v_mul_f32 v10, v10, v25\n"
      "s_add_u32 s75, s94, 1\n"
      "s_lshl_b32 s76, s94, 5\n"
      "v_readfirstlane_b32 s79, v10\n"
      "s_and_b32 s94, s75, 1\n"
      "s_lshl_b32 s94, s94, 24\n"
      "s_sub_u32 s94, 33554432, s94\n"
      "s_sub_u32 s77, s98, s94\n"
      "s_subb_u32 s78, s99, 0\n"
      "s_and_b32 s78, s78, 0xffff\n"
      "v_mov_b32 v8, s75\n"
      "global_store_dword v28, v8, s[98:99] sc1\n"
      "s_waitcnt vmcnt(16)\n"
      "buffer_load_dwordx4 a[128:131], v5, s[64:67], s85 offen\n"
      "s_add_u32 s85, s85, 262144\n"
      "buffer_load_dwordx4 a[132:135], v5, s[64:67], s85 offen\n"
      "s_add_u32 s85, s85, 262144\n"
      "buffer_load_dwordx4 a[136:139], v5, s[64:67], s85 offen\n"
      "s_add_u32 s85, s85, 262144\n"
      "buffer_load_dwordx4 a[140:143], v5, s[64:67], s85 offen\n"
      "s_add_u32 s85, s85, 262144\n"
      "buffer_load_dwordx4 a[144:147], v5, s[64:67], s85 offen\n"
      "s_add_u32 s85, s85, 262144\n"
      "buffer_load_dwordx4 a[148:151], v5, s[64:67], s85 offen\n"
      "s_add_u32 s85, s85, 262144\n"
      "buffer_load_dwordx4 a[152:155], v5, s[64:67], s85 offen\n"
      "s_add_u32 s85, s85, 262144\n"
      "buffer_load_dwordx4 a[156:159], v5, s[64:67], s85 offen\n"
      "s_add_u32 s85, s85, 262144\n"
      "s_and_b32 s85, s85, 0xffffff\n"
      "ds_write_b128 v6, a[32:35] offset:0\n"
      "ds_write_b128 v16, a[36:39] offset:1024\n"
      "ds_write_b128 v6, a[40:43] offset:2048\n"
      "ds_write_b128 v16, a[44:47] offset:3072\n"
      "ds_write_b128 v6, a[48:51] offset:4096\n"
      "ds_write_b128 v16, a[52:55] offset:5120\n"
      "ds_write_b128 v6, a[56:59] offset:6144\n"
      "ds_write_b128 v16, a[60:63] offset:7168\n"
      "ds_read_b64_tr_b8 a[32:33], v7 offset:0\n"
      "ds_read_b64_tr_b8 a[34:35], v7 offset:512\n"
      "ds_read_b64_tr_b8 a[36:37], v7 offset:4096\n"
      "ds_read_b64_tr_b8 a[38:39], v7 offset:4608\n"
      "ds_read_b64_tr_b8 a[40:41], v17 offset:0\n"
      "ds_read_b64_tr_b8 a[42:43], v17 offset:512\n"
      "ds_read_b64_tr_b8 a[44:45], v17 offset:4096\n"
      "ds_read_b64_tr_b8 a[46:47], v17 offset:4608\n"
      "ds_read_b64_tr_b8 a[48:49], v18 offset:0\n"
      "ds_read_b64_tr_b8 a[50:51], v18 offset:512\n"
      "ds_read_b64_tr_b8 a[52:53], v18 offset:4096\n"
      "ds_read_b64_tr_b8 a[54:55], v18 offset:4608\n"
      "ds_read_b64_tr_b8 a[56:57], v19 offset:0\n"
      "ds_read_b64_tr_b8 a[58:59], v19 offset:512\n"
      "ds_read_b64_tr_b8 a[60:61], v19 offset:4096\n"
      "ds_read_b64_tr_b8 a[62:63], v19 offset:4608\n"
      "s_waitcnt vmcnt(16)\n"
      "buffer_load_dwordx4 a[160:163], v5, s[64:67], s85 offen\n"
      "s_add_u32 s85, s85, 262144\n"
      "buffer_load_dwordx4 a[164:167], v5, s[64:67], s85 offen\n"
      "s_add_u32 s85, s85, 262144\n"
      "buffer_load_dwordx4 a[168:171], v5, s[64:67], s85 offen\n"
      "s_add_u32 s85, s85, 262144\n"
      "buffer_load_dwordx4 a[172:175], v5, s[64:67], s85 offen\n"
      "s_add_u32 s85, s85, 262144\n"
      "buffer_load_dwordx4 a[176:179], v5, s[64:67], s85 offen\n"
      "s_add_u32 s85, s85, 262144\n"
      "buffer_load_dwordx4 a[180:183], v5, s[64:67], s85 offen\n"
      "s_add_u32 s85, s85, 262144\n"
      "buffer_load_dwordx4 a[184:187], v5, s[64:67], s85 offen\n"
      "s_add_u32 s85, s85, 262144\n"
      "buffer_load_dwordx4 a[188:191], v5, s[64:67], s85 offen\n"
      "s_add_u32 s85, s85, 262144\n"
      "s_and_b32 s85, s85, 0xffffff\n"
      "ds_write_b128 v6, a[64:67] offset:0\n"
      "ds_write_b128 v16, a[68:71] offset:1024\n"
      "ds_write_b128 v6, a[72:75] offset:2048\n"
      "ds_write_b128 v16, a[76:79] offset:3072\n"
      "ds_write_b128 v6, a[80:83] offset:4096\n"
      "ds_write_b128 v16, a[84:87] offset:5120\n"
      "ds_write_b128 v6, a[88:91] offset:6144\n"
      "ds_write_b128 v16, a[92:95] offset:7168\n"
      "ds_read_b64_tr_b8 a[64:65], v7 offset:0\n"
      "ds_read_b64_tr_b8 a[66:67], v7 offset:512\n"
      "ds_read_b64_tr_b8 a[68:69], v7 offset:4096\n"
      "ds_read_b64_tr_b8 a[70:71], v7 offset:4608\n"
      "ds_read_b64_tr_b8 a[72:73], v17 offset:0\n"
      "ds_read_b64_tr_b8 a[74:75], v17 offset:512\n"
      "ds_read_b64_tr_b8 a[76:77], v17 offset:4096\n"
      "ds_read_b64_tr_b8 a[78:79], v17 offset:4608\n"
      "ds_read_b64_tr_b8 a[80:81], v18 offset:0\n"
      "ds_read_b64_tr_b8 a[82:83], v18 offset:512\n"
      "ds_read_b64_tr_b8 a[84:85], v18 offset:4096\n"
      "ds_read_b64_tr_b8 a[86:87], v18 offset:4608\n"
      "ds_read_b64_tr_b8 a[88:89], v19 offset:0\n"
      "ds_read_b64_tr_b8 a[90:91], v19 offset:512\n"
      "ds_read_b64_tr_b8 a[92:93], v19 offset:4096\n"
      "ds_read_b64_tr_b8 a[94:95], v19 offset:4608\n"
      "s_waitcnt vmcnt(16)\n"
      "buffer_load_dwordx4 a[192:195], v5, s[64:67], s85 offen\n"
      "s_add_u32 s85, s85, 262144\n"
      "buffer_load_dwordx4 a[196:199], v5, s[64:67], s85 offen\n"
      "s_add_u32 s85, s85, 262144\n"
      "buffer_load_dwordx4 a[200:203], v5, s[64:67], s85 offen\n"
      "s_add_u32 s85, s85, 262144\n"
      "buffer_load_dwordx4 a[204:207], v5, s[64:67], s85 offen\n"
      "s_add_u32 s85, s85, 262144\n"
      "buffer_load_dwordx4 a[208:211], v5, s[64:67], s85 offen\n"
      "s_add_u32 s85, s85, 262144\n"
      "buffer_load_dwordx4 a[212:215], v5, s[64:67], s85 offen\n"
      "s_add_u32 s85, s85, 262144\n"
      "buffer_load_dwordx4 a[216:219], v5, s[64:67], s85 offen\n"
      "s_add_u32 s85, s85, 262144\n"
      "buffer_load_dwordx4 a[220:223], v5, s[64:67], s85 offen\n"
      "s_add_u32 s85, s85, 262144\n"
      "s_and_b32 s85, s85, 0xffffff\n"
      "ds_write_b128 v6, a[96:99] offset:0\n"
      "ds_write_b128 v16, a[100:103] offset:1024\n"
      "ds_write_b128 v6, a[104:107] offset:2048\n"
      "ds_write_b128 v16, a[108:111] offset:3072\n"
      "ds_write_b128 v6, a[112:115] offset:4096\n"
      "ds_write_b128 v16, a[116:119] offset:5120\n"
      "ds_write_b128 v6, a[120:123] offset:6144\n"
      "ds_write_b128 v16, a[124:127] offset:7168\n"
      "ds_read_b64_tr_b8 a[96:97], v7 offset:0\n"
      "ds_read_b64_tr_b8 a[98:99], v7 offset:512\n"
      "ds_read_b64_tr_b8 a[100:101], v7 offset:4096\n"
      "ds_read_b64_tr_b8 a[102:103], v7 offset:4608\n"
      "ds_read_b64_tr_b8 a[104:105], v17 offset:0\n"
      "ds_read_b64_tr_b8 a[106:107], v17 offset:512\n"
      "ds_read_b64_tr_b8 a[108:109], v17 offset:4096\n"
      "ds_read_b64_tr_b8 a[110:111], v17 offset:4608\n"
      "ds_read_b64_tr_b8 a[112:113], v18 offset:0\n"
      "ds_read_b64_tr_b8 a[114:115], v18 offset:512\n"
      "ds_read_b64_tr_b8 a[116:117], v18 offset:4096\n"
      "ds_read_b64_tr_b8 a[118:119], v18 offset:4608\n"
      "ds_read_b64_tr_b8 a[120:121], v19 offset:0\n"
      "ds_read_b64_tr_b8 a[122:123], v19 offset:512\n"
      "ds_read_b64_tr_b8 a[124:125], v19 offset:4096\n"
      "ds_read_b64_tr_b8 a[126:127], v19 offset:4608\n"
      "s_waitcnt vmcnt(16)\n"
      "buffer_load_dwordx4 a[224:227], v5, s[64:67], s85 offen\n"
      "s_add_u32 s85, s85, 262144\n"
      "buffer_load_dwordx4 a[228:231], v5, s[64:67], s85 offen\n"
      "s_add_u32 s85, s85, 262144\n"
      "buffer_load_dwordx4 a[232:235], v5, s[64:67], s85 offen\n"
      "s_add_u32 s85, s85, 262144\n"
      "buffer_load_dwordx4 a[236:239], v5, s[64:67], s85 offen\n"
      "s_add_u32 s85, s85, 262144\n"
      "buffer_load_dwordx4 a[240:243], v5, s[64:67], s85 offen\n"
      "s_add_u32 s85, s85, 262144\n"
      "buffer_load_dwordx4 a[244:247], v5, s[64:67], s85 offen\n"
      "s_add_u32 s85, s85, 262144\n"
      "buffer_load_dwordx4 a[248:251], v5, s[64:67], s85 offen\n"
      "s_add_u32 s85, s85, 262144\n"
      "buffer_load_dwordx4 a[252:255], v5, s[64:67], s85 offen\n"
      "s_add_u32 s85, s85, 262144\n"
      "s_and_b32 s85, s85, 0xffffff\n"
      "ds_write_b128 v6, a[128:131] offset:0\n"
      "ds_write_b128 v16, a[132:135] offset:1024\n"
      "ds_write_b128 v6, a[136:139] offset:2048\n"
      "ds_write_b128 v16, a[140:143] offset:3072\n"
      "ds_write_b128 v6, a[144:147] offset:4096\n"
      "ds_write_b128 v16, a[148:151] offset:5120\n"
      "ds_write_b128 v6, a[152:155] offset:6144\n"
      "ds_write_b128 v16, a[156:159] offset:7168\n"
      "ds_read_b64_tr_b8 a[128:129], v7 offset:0\n"
      "ds_read_b64_tr_b8 a[130:131], v7 offset:512\n"
      "ds_read_b64_tr_b8 a[132:133], v7 offset:4096\n"
      "ds_read_b64_tr_b8 a[134:135], v7 offset:4608\n"
      "ds_read_b64_tr_b8 a[136:137], v17 offset:0\n"
      "ds_read_b64_tr_b8 a[138:139], v17 offset:512\n"
      "ds_read_b64_tr_b8 a[140:141], v17 offset:4096\n"
      "ds_read_b64_tr_b8 a[142:143], v17 offset:4608\n"
      "ds_read_b64_tr_b8 a[144:145], v18 offset:0\n"
      "ds_read_b64_tr_b8 a[146:147], v18 offset:512\n"
      "ds_read_b64_tr_b8 a[148:149], v18 offset:4096\n"
      "ds_read_b64_tr_b8 a[150:151], v18 offset:4608\n"
      "ds_read_b64_tr_b8 a[152:153], v19 offset:0\n"
      "ds_read_b64_tr_b8 a[154:155], v19 offset:512\n"
      "ds_read_b64_tr_b8 a[156:157], v19 offset:4096\n"
      "ds_read_b64_tr_b8 a[158:159], v19 offset:4608\n"
      "s_waitcnt vmcnt(16)\n"
      "ds_write_b128 v6, a[160:163] offset:0\n"
      "ds_write_b128 v16, a[164:167] offset:1024\n"
      "ds_write_b128 v6, a[168:171] offset:2048\n"
      "ds_write_b128 v16, a[172:175] offset:3072\n"
      "ds_write_b128 v6, a[176:179] offset:4096\n"
      "ds_write_b128 v16, a[180:183] offset:5120\n"
      "ds_write_b128 v6, a[184:187] offset:6144\n"
      "ds_write_b128 v16, a[188:191] offset:7168\n"
      "ds_read_b64_tr_b8 a[160:161], v7 offset:0\n"
      "ds_read_b64_tr_b8 a[162:163], v7 offset:512\n"
      "ds_read_b64_tr_b8 a[164:165], v7 offset:4096\n"
      "ds_read_b64_tr_b8 a[166:167], v7 offset:4608\n"
      "ds_read_b64_tr_b8 a[168:169], v17 offset:0\n"
      "ds_read_b64_tr_b8 a[170:171], v17 offset:512\n"
      "ds_read_b64_tr_b8 a[172:173], v17 offset:4096\n"
      "ds_read_b64_tr_b8 a[174:175], v17 offset:4608\n"
      "ds_read_b64_tr_b8 a[176:177], v18 offset:0\n"
      "ds_read_b64_tr_b8 a[178:179], v18 offset:512\n"
      "ds_read_b64_tr_b8 a[180:181], v18 offset:4096\n"
      "ds_read_b64_tr_b8 a[182:183], v18 offset:4608\n"
      "ds_read_b64_tr_b8 a[184:185], v19 offset:0\n"
      "ds_read_b64_tr_b8 a[186:187], v19 offset:512\n"
      "ds_read_b64_tr_b8 a[188:189], v19 offset:4096\n"
      "ds_read_b64_tr_b8 a[190:191], v19 offset:4608\n"
      "s_waitcnt vmcnt(8)\n"
      "ds_write_b128 v6, a[192:195] offset:0\n"
      "ds_write_b128 v16, a[196:199] offset:1024\n"
      "ds_write_b128 v6, a[200:203] offset:2048\n"
      "ds_write_b128 v16, a[204:207] offset:3072\n"
      "ds_write_b128 v6, a[208:211] offset:4096\n"
      "ds_write_b128 v16, a[212:215] offset:5120\n"
      "ds_write_b128 v6, a[216:219] offset:6144\n"
      "ds_write_b128 v16, a[220:223] offset:7168\n"
      "ds_read_b64_tr_b8 a[192:193], v7 offset:0\n"
      "ds_read_b64_tr_b8 a[194:195], v7 offset:512\n"
      "ds_read_b64_tr_b8 a[196:197], v7 offset:4096\n"
      "ds_read_b64_tr_b8 a[198:199], v7 offset:4608\n"
      "ds_read_b64_tr_b8 a[200:201], v17 offset:0\n"
      "ds_read_b64_tr_b8 a[202:203], v17 offset:512\n"
      "ds_read_b64_tr_b8 a[204:205], v17 offset:4096\n"
      "ds_read_b64_tr_b8 a[206:207], v17 offset:4608\n"
      "ds_read_b64_tr_b8 a[208:209], v18 offset:0\n"
      "ds_read_b64_tr_b8 a[210:211], v18 offset:512\n"
      "ds_read_b64_tr_b8 a[212:213], v18 offset:4096\n"
      "ds_read_b64_tr_b8 a[214:215], v18 offset:4608\n"
      "ds_read_b64_tr_b8 a[216:217], v19 offset:0\n"
      "ds_read_b64_tr_b8 a[218:219], v19 offset:512\n"
      "ds_read_b64_tr_b8 a[220:221], v19 offset:4096\n"
      "ds_read_b64_tr_b8 a[222:223], v19 offset:4608\n"
      "s_waitcnt vmcnt(0)\n"
      "ds_write_b128 v6, a[224:227] offset:0\n"
      "ds_write_b128 v16, a[228:231] offset:1024\n"
      "ds_write_b128 v6, a[232:235] offset:2048\n"
      "ds_write_b128 v16, a[236:239] offset:3072\n"
      "ds_write_b128 v6, a[240:243] offset:4096\n"
      "ds_write_b128 v16, a[244:247] offset:5120\n"
      "ds_write_b128 v6, a[248:251] offset:6144\n"
      "ds_write_b128 v16, a[252:255] offset:7168\n"
      "ds_read_b64_tr_b8 a[224:225], v7 offset:0\n"
      "ds_read_b64_tr_b8 a[226:227], v7 offset:512\n"
      "ds_read_b64_tr_b8 a[228:229], v7 offset:4096\n"
      "ds_read_b64_tr_b8 a[230:231], v7 offset:4608\n"
      "ds_read_b64_tr_b8 a[232:233], v17 offset:0\n"
      "ds_read_b64_tr_b8 a[234:235], v17 offset:512\n"
      "ds_read_b64_tr_b8 a[236:237], v17 offset:4096\n"
      "ds_read_b64_tr_b8 a[238:239], v17 offset:4608\n"
      "ds_read_b64_tr_b8 a[240:241], v18 offset:0\n"
      "ds_read_b64_tr_b8 a[242:243], v18 offset:512\n"
      "ds_read_b64_tr_b8 a[244:245], v18 offset:4096\n"
      "ds_read_b64_tr_b8 a[246:247], v18 offset:4608\n"
      "ds_read_b64_tr_b8 a[248:249], v19 offset:0\n"
      "ds_read_b64_tr_b8 a[250:251], v19 offset:512\n"
      "ds_read_b64_tr_b8 a[252:253], v19 offset:4096\n"
      "ds_read_b64_tr_b8 a[254:255], v19 offset:4608\n"
      "v_xor_b32 v5, 1024, v3\n"
      "v_xor_b32 v6, 2048, v3\n"
      "v_xor_b32 v7, 3072, v3\n"
      "s_waitcnt vmcnt(0) lgkmcnt(0)\n"
      "s_mov_b32 s40, 4\n"
      "s_mov_b32 s41, 0\n"
      "v_mov_b32 v8, 0\n"
      "s_branch .Lslow0_%=\n"
      ".p2align 6\n"
      ".Lgo0_%=:\n"
      "s_waitcnt vmcnt(14)\n"
      "v_mfma_f32_16x16x128_f8f6f4 v[16:19], a[0:7], v[64:71], 0 cbsz:1\n"
      "s_add_u32 s94, s72, 1\n"
      "s_lshl_b32 s95, s44, 16\n"
      "s_add_u32 s64, s77, s95\n"
      "s_addc_u32 s65, s78, 0\n"
      "s_cmp_ge_u32 s94, s47\n"
      "v_mfma_f32_16x16x128_f8f6f4 v[20:23], a[8:15], v[64:71], 0 cbsz:1\n"
      "s_cselect_b32 s46, 1, 0\n"
      "s_cmp_lt_u32 s94, %[nb]\n"
      "s_cselect_b32 s46, s46, 0\n"
      "s_cmp_gt_u32 s74, s94\n"
      "s_cselect_b32 s97, 1, 0\n"
      "v_mfma_f32_16x16x128_f8f6f4 v[24:27], a[16:23], v[64:71], 0 cbsz:1\n"
      "s_cselect_b32 s96, 1, s46\n"
      "s_cselect_b32 s46, 0, s46\n"
      "s_mul_i32 s66, s96, 65536\n"
      "s_add_u32 s73, s73, s97\n"
      "buffer_load_dwordx4 v[128:131], v1, s[64:67], s48 offen\n"
      "v_mfma_f32_16x16x128_f8f6f4 v[28:31], a[24:31], v[64:71], 0 cbsz:1\n"
      "s_add_u32 s94, s72, 2\n"
      "s_mul_i32 s96, s94, 7\n"
      "s_add_u32 s96, s96, %[ownu]\n"
      "s_add_u32 s95, s94, %[srcu]\n"
      "s_cmp_lt_u32 s94, %[nownr]\n"
      "s_cselect_b32 s45, s96, s95\n"
      "s_add_u32 s95, s95, %[nown]\n"
      "s_cmp_lt_u32 s94, 4\n"
      "s_cselect_b32 s45, s95, s45\n"
      "buffer_load_dwordx4 v[132:135], v1, s[64:67], s49 offen\n"
      "s_waitcnt vmcnt(14)\n"
      "v_mfma_f32_16x16x128_f8f6f4 v[16:19], a[32:39], v[72:79], v[16:19] cbsz:1\n"
      "buffer_load_dwordx4 v[136:139], v1, s[64:67], s50 offen\n"
      "v_mfma_f32_16x16x128_f8f6f4 v[20:23], a[40:47], v[72:79], v[20:23] cbsz:1\n"
      "s_lshl_b32 s85, s42, 17\n"
      "s_cmp_lg_u32 s72, 0\n"
      "s_cselect_b32 s70, 33554432, 0\n"
      "s_mov_b32 s42, s43\n"
      "s_mov_b32 s43, s44\n"
      "s_mov_b32 s44, s45\n"
      "v_mfma_f32_16x16x128_f8f6f4 v[24:27], a[48:55], v[72:79], v[24:27] cbsz:1\n"
      "buffer_load_dwordx4 v[140:143], v1, s[64:67], s51 offen\n"
      "v_mfma_f32_16x16x128_f8f6f4 v[28:31], a[56:63], v[72:79], v[28:31] cbsz:1\n"
      "ds_write_b128 v3, v[52:55] offset:24576\n"
      "s_waitcnt vmcnt(14)\n"
      "v_mfma_f32_16x16x128_f8f6f4 v[16:19], a[64:71], v[80:87], v[16:19] cbsz:1\n"
      "buffer_load_dwordx4 v[144:147], v1, s[64:67], s52 offen\n"
      "ds_write_b128 v3, v[56:59] offset:28672\n"
      "v_mfma_f32_16x16x128_f8f6f4 v[20:23], a[72:79], v[80:87], v[20:23] cbsz:1\n"
      "ds_write_b128 v3, v[60:63] offset:32768\n"
      "v_mfma_f32_16x16x128_f8f6f4 v[24:27], a[80:87], v[80:87], v[24:27] cbsz:1\n"
      "s_add_u32 s94, s72, 3\n"
      "s_min_u32 s94, s94, %[nb]\n"
      "s_cmp_lt_u32 s74, s94\n"
      "s_cselect_b32 s38, 4096, 0\n"
      "buffer_load_dwordx4 v[148:151], v1, s[64:67], s53 offen\n"
      "v_mfma_f32_16x16x128_f8f6f4 v[28:31], a[88:95], v[80:87], v[28:31] cbsz:1\n"
      "buffer_load_dword v8, v4, s[36:39], %[po] offen sc1\n"
      "s_waitcnt vmcnt(15)\n"
      "v_mfma_f32_16x16x128_f8f6f4 v[16:19], a[96:103], v[88:95], v[16:19] cbsz:1\n"
      "buffer_load_dwordx4 v[152:155], v1, s[64:67], s54 offen\n"
      "v_mfma_f32_16x16x128_f8f6f4 v[20:23], a[104:111], v[88:95], v[20:23] cbsz:1\n"
      "v_mfma_f32_16x16x128_f8f6f4 v[24:27], a[112:119], v[88:95], v[24:27] cbsz:1\n"
      "buffer_load_dwordx4 v[156:159], v1, s[64:67], s55 offen\n"
      "s_cmp_lg_u32 s41, 0\n"
      "s_cbranch_scc1 .Lmid0_%=\n"
      ".Lmidret0_%=:\n"
      "v_mfma_f32_16x16x128_f8f6f4 v[28:31], a[120:127], v[88:95], v[28:31] cbsz:1\n"
      "s_waitcnt lgkmcnt(0)\n"
      "s_barrier\n"
      "ds_read_b128 v[52:55], v5 offset:24576\n"
      "ds_read_b128 v[56:59], v6 offset:28672\n"
      "ds_read_b128 v[60:63], v7 offset:32768\n"
      "s_waitcnt vmcnt(15)\n"
      "v_mfma_f32_16x16x128_f8f6f4 v[16:19], a[128:135], v[96:103], v[16:19] cbsz:1\n"
      "buffer_load_dwordx4 v[160:163], v1, s[64:67], s56 offen\n"
      "v_mfma_f32_16x16x128_f8f6f4 v[20:23], a[136:143], v[96:103], v[20:23] cbsz:1\n"
      "v_mfma_f32_16x16x128_f8f6f4 v[24:27], a[144:151], v[96:103], v[24:27] cbsz:1\n"
      "buffer_load_dwordx4 v[164:167], v1, s[64:67], s57 offen\n"
      "v_mfma_f32_16x16x128_f8f6f4 v[28:31], a[152:159], v[96:103], v[28:31] cbsz:1\n"
      "s_waitcnt lgkmcnt(0)\n"
      "v_add_f32 v48, v48, v52\n"
      "v_add_f32 v49, v49, v53\n"
      "v_add_f32 v50, v50, v54\n"
      "v_add_f32 v51, v51, v55\n"
      "s_waitcnt vmcnt(15)\n"
      "v_mfma_f32_16x16x128_f8f6f4 v[16:19], a[160:167], v[104:111], v[16:19] cbsz:1\n"
      "buffer_load_dwordx4 v[168:171], v1, s[64:67], s58 offen\n"
      "v_add_f32 v56, v56, v60\n"
      "v_add_f32 v57, v57, v61\n"
      "v_add_f32 v58, v58, v62\n"
      "v_add_f32 v59, v59, v63\n"
      "v_mfma_f32_16x16x128_f8f6f4 v[20:23], a[168:175], v[104:111], v[20:23] cbsz:1\n"
      "v_add_f32 v48, v48, v56\n"
      "v_add_f32 v49, v49, v57\n"
      "v_add_f32 v50, v50, v58\n"
      "v_add_f32 v51, v51, v59\n"
      "v_mfma_f32_16x16x128_f8f6f4 v[24:27], a[176:183], v[104:111], v[24:27] cbsz:1\n"
      "buffer_load_dwordx4 v[172:175], v1, s[64:67], s59 offen\n"
      "v_mul_f32 v48, s79, v48\n"
      "v_mul_f32 v49, s79, v49\n"
      "v_mul_f32 v50, s79, v50\n"
      "v_mul_f32 v51, s79, v51\n"
      "v_mfma_f32_16x16x128_f8f6f4 v[28:31], a[184:191], v[104:111], v[28:31] cbsz:1\n"
      "v_cmp_gt_f32 s[86:87], 0, v48\n"
      "v_cmp_gt_f32 s[88:89], 0, v49\n"
      "v_cmp_gt_f32 s[90:91], 0, v50\n"
      "v_cmp_gt_f32 s[92:93], 0, v51\n"
      "s_waitcnt vmcnt(15)\n"
      "v_mfma_f32_16x16x128_f8f6f4 v[16:19], a[192:199], v[112:119], v[16:19] cbsz:1\n"
      "buffer_load_dwordx4 v[176:179], v1, s[64:67], s60 offen\n"
      "v_cndmask_b32 v48, v48, 0, s[86:87]\n"
      "v_cndmask_b32 v49, v49, 0, s[88:89]\n"
      "v_cndmask_b32 v50, v50, 0, s[90:91]\n"
      "v_cndmask_b32 v51, v51, 0, s[92:93]\n"
      "v_mfma_f32_16x16x128_f8f6f4 v[20:23], a[200:207], v[112:119], v[20:23] cbsz:1\n"
      "buffer_store_dwordx4 v[48:51], v2, s[68:71], s85 offen nt\n"
      "v_mfma_f32_16x16x128_f8f6f4 v[24:27], a[208:215], v[112:119], v[24:27] cbsz:1\n"
      "buffer_load_dwordx4 v[180:183], v1, s[64:67], s61 offen\n"
      "v_mfma_f32_16x16x128_f8f6f4 v[28:31], a[216:223], v[112:119], v[28:31] cbsz:1\n"
      "s_waitcnt vmcnt(9)\n"
      "v_min_u32_dpp v9, v8, v8 quad_perm:[1,0,3,2] row_mask:0xf bank_mask:0xf\n"
      "s_waitcnt vmcnt(16)\n"
      "v_mfma_f32_16x16x128_f8f6f4 v[16:19], a[224:231], v[120:127], v[16:19] cbsz:1\n"
      "s_nop 0\n"
      "v_min_u32_dpp v9, v9, v9 quad_perm:[2,3,0,1] row_mask:0xf bank_mask:0xf\n"
      "buffer_load_dwordx4 v[184:187], v1, s[64:67], s62 offen\n"
      "v_mfma_f32_16x16x128_f8f6f4 v[20:23], a[232:239], v[120:127], v[20:23] cbsz:1\n"
      "v_readfirstlane_b32 s94, v9\n"
      "s_sub_u32 s94, s94, s76\n"
      "s_min_i32 s94, s94, 32\n"
      "s_cmp_ge_i32 s94, 4\n"
      "buffer_load_dwordx4 v[188:191], v1, s[64:67], s63 offen\n"
      "v_mfma_f32_16x16x128_f8f6f4 v[24:27], a[240:247], v[120:127], v[24:27] cbsz:1\n"
      "s_cselect_b32 s95, %[nown], 0\n"
      "s_add_i32 s94, s94, s95\n"
      "s_min_i32 s94, s94, s40\n"
      "s_max_i32 s74, s74, s94\n"
      "v_mfma_f32_16x16x128_f8f6f4 v[28:31], a[248:255], v[120:127], v[28:31] cbsz:1\n"
      "s_add_u32 s72, s72, 1\n"
      "s_cmp_le_u32 s73, s72\n"
      "s_cbranch_scc1 .Lslow1_%=\n"
      ".Lgo1_%=:\n"
      "s_waitcnt vmcnt(14)\n"
      "v_mfma_f32_16x16x128_f8f6f4 v[32:35], a[0:7], v[128:135], 0 cbsz:1\n"
      "s_add_u32 s94, s72, 1\n"
      "s_lshl_b32 s95, s44, 16\n"
      "s_add_u32 s64, s77, s95\n"
      "s_addc_u32 s65, s78, 0\n"
      "s_cmp_ge_u32 s94, s47\n"
      "v_mfma_f32_16x16x128_f8f6f4 v[36:39], a[8:15], v[128:135], 0 cbsz:1\n"
      "s_cselect_b32 s46, 1, 0\n"
      "s_cmp_lt_u32 s94, %[nb]\n"
      "s_cselect_b32 s46, s46, 0\n"
      "s_cmp_gt_u32 s74, s94\n"
      "s_cselect_b32 s97, 1, 0\n"
      "v_mfma_f32_16x16x128_f8f6f4 v[40:43], a[16:23], v[128:135], 0 cbsz:1\n"
      "s_cselect_b32 s96, 1, s46\n"
      "s_cselect_b32 s46, 0, s46\n"
      "s_mul_i32 s66, s96, 65536\n"
      "s_add_u32 s73, s73, s97\n"
      "buffer_load_dwordx4 v[192:195], v1, s[64:67], s48 offen\n"
      "v_mfma_f32_16x16x128_f8f6f4 v[44:47], a[24:31], v[128:135], 0 cbsz:1\n"
      "s_add_u32 s94, s72, 2\n"
      "s_mul_i32 s96, s94, 7\n"
      "s_add_u32 s96, s96, %[ownu]\n"
      "s_add_u32 s95, s94, %[srcu]\n"
      "s_cmp_lt_u32 s94, %[nownr]\n"
      "s_cselect_b32 s45, s96, s95\n"
      "s_add_u32 s95, s95, %[nown]\n"
      "s_cmp_lt_u32 s94, 4\n"
      "s_cselect_b32 s45, s95, s45\n"
      "buffer_load_dwordx4 v[196:199], v1, s[64:67], s49 offen\n"
      "s_waitcnt vmcnt(14)\n"
      "v_mfma_f32_16x16x128_f8f6f4 v[32:35], a[32:39], v[136:143], v[32:35] cbsz:1\n"
      "buffer_load_dwordx4 v[200:203], v1, s[64:67], s50 offen\n"
      "v_mfma_f32_16x16x128_f8f6f4 v[36:39], a[40:47], v[136:143], v[36:39] cbsz:1\n"
      "s_lshl_b32 s85, s42, 17\n"
      "s_cmp_lg_u32 s72, 0\n"
      "s_cselect_b32 s70, 33554432, 0\n"
      "s_mov_b32 s42, s43\n"
      "s_mov_b32 s43, s44\n"
      "s_mov_b32 s44, s45\n"
      "v_mfma_f32_16x16x128_f8f6f4 v[40:43], a[48:55], v[136:143], v[40:43] cbsz:1\n"
      "buffer_load_dwordx4 v[204:207], v1, s[64:67], s51 offen\n"
      "v_mfma_f32_16x16x128_f8f6f4 v[44:47], a[56:63], v[136:143], v[44:47] cbsz:1\n"
      "ds_write_b128 v3, v[20:23] offset:0\n"
      "s_waitcnt vmcnt(14)\n"
      "v_mfma_f32_16x16x128_f8f6f4 v[32:35], a[64:71], v[144:151], v[32:35] cbsz:1\n"
      "buffer_load_dwordx4 v[208:211], v1, s[64:67], s52 offen\n"
      "ds_write_b128 v3, v[24:27] offset:4096\n"
      "v_mfma_f32_16x16x128_f8f6f4 v[36:39], a[72:79], v[144:151], v[36:39] cbsz:1\n"
      "ds_write_b128 v3, v[28:31] offset:8192\n"
      "v_mfma_f32_16x16x128_f8f6f4 v[40:43], a[80:87], v[144:151], v[40:43] cbsz:1\n"
      "s_add_u32 s94, s72, 3\n"
      "s_min_u32 s94, s94, %[nb]\n"
      "s_cmp_lt_u32 s74, s94\n"
      "s_cselect_b32 s38, 4096, 0\n"
      "buffer_load_dwordx4 v[212:215], v1, s[64:67], s53 offen\n"
      "v_mfma_f32_16x16x128_f8f6f4 v[44:47], a[88:95], v[144:151], v[44:47] cbsz:1\n"
      "buffer_load_dword v8, v4, s[36:39], %[po] offen sc1\n"
      "s_waitcnt vmcnt(15)\n"
      "v_mfma_f32_16x16x128_f8f6f4 v[32:35], a[96:103], v[152:159], v[32:35] cbsz:1\n"
      "buffer_load_dwordx4 v[216:219], v1, s[64:67], s54 offen\n"
      "v_mfma_f32_16x16x128_f8f6f4 v[36:39], a[104:111], v[152:159], v[36:39] cbsz:1\n"
      "v_mfma_f32_16x16x128_f8f6f4 v[40:43], a[112:119], v[152:159], v[40:43] cbsz:1\n"
      "buffer_load_dwordx4 v[220:223], v1, s[64:67], s55 offen\n"
      "s_cmp_lg_u32 s41, 0\n"
      "s_cbranch_scc1 .Lmid1_%=\n"
      ".Lmidret1_%=:\n"
      "v_mfma_f32_16x16x128_f8f6f4 v[44:47], a[120:127], v[152:159], v[44:47] cbsz:1\n"
      "s_waitcnt lgkmcnt(0)\n"
      "s_barrier\n"
      "ds_read_b128 v[20:23], v5 offset:0\n"
      "ds_read_b128 v[24:27], v6 offset:4096\n"
      "ds_read_b128 v[28:31], v7 offset:8192\n"
      "s_waitcnt vmcnt(15)\n"
      "v_mfma_f32_16x16x128_f8f6f4 v[32:35], a[128:135], v[160:167], v[32:35] cbsz:1\n"
      "buffer_load_dwordx4 v[224:227], v1, s[64:67], s56 offen\n"
      "v_mfma_f32_16x16x128_f8f6f4 v[36:39], a[136:143], v[160:167], v[36:39] cbsz:1\n"
      "v_mfma_f32_16x16x128_f8f6f4 v[40:43], a[144:151], v[160:167], v[40:43] cbsz:1\n"
      "buffer_load_dwordx4 v[228:231], v1, s[64:67], s57 offen\n"
      "v_mfma_f32_16x16x128_f8f6f4 v[44:47], a[152:159], v[160:167], v[44:47] cbsz:1\n"
      "s_waitcnt lgkmcnt(0)\n"
      "v_add_f32 v16, v16, v20\n"
      "v_add_f32 v17, v17, v21\n"
      "v_add_f32 v18, v18, v22\n"
      "v_add_f32 v19, v19, v23\n"
      "s_waitcnt vmcnt(15)\n"
      "v_mfma_f32_16x16x128_f8f6f4 v[32:35], a[160:167], v[168:175], v[32:35] cbsz:1\n"
      "buffer_load_dwordx4 v[232:235], v1, s[64:67], s58 offen\n"
      "v_add_f32 v24, v24, v28\n"
      "v_add_f32 v25, v25, v29\n"
      "v_add_f32 v26, v26, v30\n"
      "v_add_f32 v27, v27, v31\n"
      "v_mfma_f32_16x16x128_f8f6f4 v[36:39], a[168:175], v[168:175], v[36:39] cbsz:1\n"
      "v_add_f32 v16, v16, v24\n"
      "v_add_f32 v17, v17, v25\n"
      "v_add_f32 v18, v18, v26\n"
      "v_add_f32 v19, v19, v27\n"
      "v_mfma_f32_16x16x128_f8f6f4 v[40:43], a[176:183], v[168:175], v[40:43] cbsz:1\n"
      "buffer_load_dwordx4 v[236:239], v1, s[64:67], s59 offen\n"
      "v_mul_f32 v16, s79, v16\n"
      "v_mul_f32 v17, s79, v17\n"
      "v_mul_f32 v18, s79, v18\n"
      "v_mul_f32 v19, s79, v19\n"
      "v_mfma_f32_16x16x128_f8f6f4 v[44:47], a[184:191], v[168:175], v[44:47] cbsz:1\n"
      "v_cmp_gt_f32 s[86:87], 0, v16\n"
      "v_cmp_gt_f32 s[88:89], 0, v17\n"
      "v_cmp_gt_f32 s[90:91], 0, v18\n"
      "v_cmp_gt_f32 s[92:93], 0, v19\n"
      "s_waitcnt vmcnt(15)\n"
      "v_mfma_f32_16x16x128_f8f6f4 v[32:35], a[192:199], v[176:183], v[32:35] cbsz:1\n"
      "buffer_load_dwordx4 v[240:243], v1, s[64:67], s60 offen\n"
      "v_cndmask_b32 v16, v16, 0, s[86:87]\n"
      "v_cndmask_b32 v17, v17, 0, s[88:89]\n"
      "v_cndmask_b32 v18, v18, 0, s[90:91]\n"
      "v_cndmask_b32 v19, v19, 0, s[92:93]\n"
      "v_mfma_f32_16x16x128_f8f6f4 v[36:39], a[200:207], v[176:183], v[36:39] cbsz:1\n"
      "buffer_store_dwordx4 v[16:19], v2, s[68:71], s85 offen nt\n"
      "v_mfma_f32_16x16x128_f8f6f4 v[40:43], a[208:215], v[176:183], v[40:43] cbsz:1\n"
      "buffer_load_dwordx4 v[244:247], v1, s[64:67], s61 offen\n"
      "v_mfma_f32_16x16x128_f8f6f4 v[44:47], a[216:223], v[176:183], v[44:47] cbsz:1\n"
      "s_waitcnt vmcnt(9)\n"
      "v_min_u32_dpp v9, v8, v8 quad_perm:[1,0,3,2] row_mask:0xf bank_mask:0xf\n"
      "s_waitcnt vmcnt(16)\n"
      "v_mfma_f32_16x16x128_f8f6f4 v[32:35], a[224:231], v[184:191], v[32:35] cbsz:1\n"
      "s_nop 0\n"
      "v_min_u32_dpp v9, v9, v9 quad_perm:[2,3,0,1] row_mask:0xf bank_mask:0xf\n"
      "buffer_load_dwordx4 v[248:251], v1, s[64:67], s62 offen\n"
      "v_mfma_f32_16x16x128_f8f6f4 v[36:39], a[232:239], v[184:191], v[36:39] cbsz:1\n"
      "v_readfirstlane_b32 s94, v9\n"
      "s_sub_u32 s94, s94, s76\n"
      "s_min_i32 s94, s94, 32\n"
      "s_cmp_ge_i32 s94, 4\n"
      "buffer_load_dwordx4 v[252:255], v1, s[64:67], s63 offen\n"
      "v_mfma_f32_16x16x128_f8f6f4 v[40:43], a[240:247], v[184:191], v[40:43] cbsz:1\n"
      "s_cselect_b32 s95, %[nown], 0\n"
      "s_add_i32 s94, s94, s95\n"
      "s_min_i32 s94, s94, s40\n"
      "s_max_i32 s74, s74, s94\n"
      "v_mfma_f32_16x16x128_f8f6f4 v[44:47], a[248:255], v[184:191], v[44:47] cbsz:1\n"
      "s_add_u32 s72, s72, 1\n"
      "s_cmp_le_u32 s73, s72\n"
      "s_cbranch_scc1 .Lslow2_%=\n"
      ".Lgo2_%=:\n"
      "s_waitcnt vmcnt(14)\n"
      "v_mfma_f32_16x16x128_f8f6f4 v[48:51], a[0:7], v[192:199], 0 cbsz:1\n"
      "s_add_u32 s94, s72, 1\n"
      "s_lshl_b32 s95, s44, 16\n"
      "s_add_u32 s64, s77, s95\n"
      "s_addc_u32 s65, s78, 0\n"
      "s_cmp_ge_u32 s94, s47\n"
      "v_mfma_f32_16x16x128_f8f6f4 v[52:55], a[8:15], v[192:199], 0 cbsz:1\n"
      "s_cselect_b32 s46, 1, 0\n"
      "s_cmp_lt_u32 s94, %[nb]\n"
      "s_cselect_b32 s46, s46, 0\n"
      "s_cmp_gt_u32 s74, s94\n"
      "s_cselect_b32 s97, 1, 0\n"
      "v_mfma_f32_16x16x128_f8f6f4 v[56:59], a[16:23], v[192:199], 0 cbsz:1\n"
      "s_cselect_b32 s96, 1, s46\n"
      "s_cselect_b32 s46, 0, s46\n"
      "s_mul_i32 s66, s96, 65536\n"
      "s_add_u32 s73, s73, s97\n"
      "buffer_load_dwordx4 v[64:67], v1, s[64:67], s48 offen\n"
      "v_mfma_f32_16x16x128_f8f6f4 v[60:63], a[24:31], v[192:199], 0 cbsz:1\n"
      "s_add_u32 s94, s72, 2\n"
      "s_mul_i32 s96, s94, 7\n"
      "s_add_u32 s96, s96, %[ownu]\n"
      "s_add_u32 s95, s94, %[srcu]\n"
      "s_cmp_lt_u32 s94, %[nownr]\n"
      "s_cselect_b32 s45, s96, s95\n"
      "s_add_u32 s95, s95, %[nown]\n"
      "s_cmp_lt_u32 s94, 4\n"
      "s_cselect_b32 s45, s95, s45\n"
      "buffer_load_dwordx4 v[68:71], v1, s[64:67], s49 offen\n"
      "s_waitcnt vmcnt(14)\n"
      "v_mfma_f32_16x16x128_f8f6f4 v[48:51], a[32:39], v[200:207], v[48:51] cbsz:1\n"
      "buffer_load_dwordx4 v[72:75], v1, s[64:67], s50 offen\n"
      "v_mfma_f32_16x16x128_f8f6f4 v[52:55], a[40:47], v[200:207], v[52:55] cbsz:1\n"
      "s_lshl_b32 s85, s42, 17\n"
      "s_cmp_lg_u32 s72, 0\n"
      "s_cselect_b32 s70, 33554432, 0\n"
      "s_mov_b32 s42, s43\n"
      "s_mov_b32 s43, s44\n"
      "s_mov_b32 s44, s45\n"
      "v_mfma_f32_16x16x128_f8f6f4 v[56:59], a[48:55], v[200:207], v[56:59] cbsz:1\n"
      "buffer_load_dwordx4 v[76:79], v1, s[64:67], s51 offen\n"
      "v_mfma_f32_16x16x128_f8f6f4 v[60:63], a[56:63], v[200:207], v[60:63] cbsz:1\n"
      "ds_write_b128 v3, v[36:39] offset:12288\n"
      "s_waitcnt vmcnt(14)\n"
      "v_mfma_f32_16x16x128_f8f6f4 v[48:51], a[64:71], v[208:215], v[48:51] cbsz:1\n"
      "buffer_load_dwordx4 v[80:83], v1, s[64:67], s52 offen\n"
      "ds_write_b128 v3, v[40:43] offset:16384\n"
      "v_mfma_f32_16x16x128_f8f6f4 v[52:55], a[72:79], v[208:215], v[52:55] cbsz:1\n"
      "ds_write_b128 v3, v[44:47] offset:20480\n"
      "v_mfma_f32_16x16x128_f8f6f4 v[56:59], a[80:87], v[208:215], v[56:59] cbsz:1\n"
      "s_add_u32 s94, s72, 3\n"
      "s_min_u32 s94, s94, %[nb]\n"
      "s_cmp_lt_u32 s74, s94\n"
      "s_cselect_b32 s38, 4096, 0\n"
      "buffer_load_dwordx4 v[84:87], v1, s[64:67], s53 offen\n"
      "v_mfma_f32_16x16x128_f8f6f4 v[60:63], a[88:95], v[208:215], v[60:63] cbsz:1\n"
      "buffer_load_dword v8, v4, s[36:39], %[po] offen sc1\n"
      "s_waitcnt vmcnt(15)\n"
      "v_mfma_f32_16x16x128_f8f6f4 v[48:51], a[96:103], v[216:223], v[48:51] cbsz:1\n"
      "buffer_load_dwordx4 v[88:91], v1, s[64:67], s54 offen\n"
      "v_mfma_f32_16x16x128_f8f6f4 v[52:55], a[104:111], v[216:223], v[52:55] cbsz:1\n"
      "v_mfma_f32_16x16x128_f8f6f4 v[56:59], a[112:119], v[216:223], v[56:59] cbsz:1\n"
      "buffer_load_dwordx4 v[92:95], v1, s[64:67], s55 offen\n"
      "s_cmp_lg_u32 s41, 0\n"
      "s_cbranch_scc1 .Lmid2_%=\n"
      ".Lmidret2_%=:\n"
      "v_mfma_f32_16x16x128_f8f6f4 v[60:63], a[120:127], v[216:223], v[60:63] cbsz:1\n"
      "s_waitcnt lgkmcnt(0)\n"
      "s_barrier\n"
      "ds_read_b128 v[36:39], v5 offset:12288\n"
      "ds_read_b128 v[40:43], v6 offset:16384\n"
      "ds_read_b128 v[44:47], v7 offset:20480\n"
      "s_waitcnt vmcnt(15)\n"
      "v_mfma_f32_16x16x128_f8f6f4 v[48:51], a[128:135], v[224:231], v[48:51] cbsz:1\n"
      "buffer_load_dwordx4 v[96:99], v1, s[64:67], s56 offen\n"
      "v_mfma_f32_16x16x128_f8f6f4 v[52:55], a[136:143], v[224:231], v[52:55] cbsz:1\n"
      "v_mfma_f32_16x16x128_f8f6f4 v[56:59], a[144:151], v[224:231], v[56:59] cbsz:1\n"
      "buffer_load_dwordx4 v[100:103], v1, s[64:67], s57 offen\n"
      "v_mfma_f32_16x16x128_f8f6f4 v[60:63], a[152:159], v[224:231], v[60:63] cbsz:1\n"
      "s_waitcnt lgkmcnt(0)\n"
      "v_add_f32 v32, v32, v36\n"
      "v_add_f32 v33, v33, v37\n"
      "v_add_f32 v34, v34, v38\n"
      "v_add_f32 v35, v35, v39\n"
      "s_waitcnt vmcnt(15)\n"
      "v_mfma_f32_16x16x128_f8f6f4 v[48:51], a[160:167], v[232:239], v[48:51] cbsz:1\n"
      "buffer_load_dwordx4 v[104:107], v1, s[64:67], s58 offen\n"
      "v_add_f32 v40, v40, v44\n"
      "v_add_f32 v41, v41, v45\n"
      "v_add_f32 v42, v42, v46\n"
      "v_add_f32 v43, v43, v47\n"
      "v_mfma_f32_16x16x128_f8f6f4 v[52:55], a[168:175], v[232:239], v[52:55] cbsz:1\n"
      "v_add_f32 v32, v32, v40\n"
      "v_add_f32 v33, v33, v41\n"
      "v_add_f32 v34, v34, v42\n"
      "v_add_f32 v35, v35, v43\n"
      "v_mfma_f32_16x16x128_f8f6f4 v[56:59], a[176:183], v[232:239], v[56:59] cbsz:1\n"
      "buffer_load_dwordx4 v[108:111], v1, s[64:67], s59 offen\n"
      "v_mul_f32 v32, s79, v32\n"
      "v_mul_f32 v33, s79, v33\n"
      "v_mul_f32 v34, s79, v34\n"
      "v_mul_f32 v35, s79, v35\n"
      "v_mfma_f32_16x16x128_f8f6f4 v[60:63], a[184:191], v[232:239], v[60:63] cbsz:1\n"
      "v_cmp_gt_f32 s[86:87], 0, v32\n"
      "v_cmp_gt_f32 s[88:89], 0, v33\n"
      "v_cmp_gt_f32 s[90:91], 0, v34\n"
      "v_cmp_gt_f32 s[92:93], 0, v35\n"
      "s_waitcnt vmcnt(15)\n"
      "v_mfma_f32_16x16x128_f8f6f4 v[48:51], a[192:199], v[240:247], v[48:51] cbsz:1\n"
      "buffer_load_dwordx4 v[112:115], v1, s[64:67], s60 offen\n"
      "v_cndmask_b32 v32, v32, 0, s[86:87]\n"
      "v_cndmask_b32 v33, v33, 0, s[88:89]\n"
      "v_cndmask_b32 v34, v34, 0, s[90:91]\n"
      "v_cndmask_b32 v35, v35, 0, s[92:93]\n"
      "v_mfma_f32_16x16x128_f8f6f4 v[52:55], a[200:207], v[240:247], v[52:55] cbsz:1\n"
      "buffer_store_dwordx4 v[32:35], v2, s[68:71], s85 offen nt\n"
      "v_mfma_f32_16x16x128_f8f6f4 v[56:59], a[208:215], v[240:247], v[56:59] cbsz:1\n"
      "buffer_load_dwordx4 v[116:119], v1, s[64:67], s61 offen\n"
      "v_mfma_f32_16x16x128_f8f6f4 v[60:63], a[216:223], v[240:247], v[60:63] cbsz:1\n"
      "s_waitcnt vmcnt(9)\n"
      "v_min_u32_dpp v9, v8, v8 quad_perm:[1,0,3,2] row_mask:0xf bank_mask:0xf\n"
      "s_waitcnt vmcnt(16)\n"
      "v_mfma_f32_16x16x128_f8f6f4 v[48:51], a[224:231], v[248:255], v[48:51] cbsz:1\n"
      "s_nop 0\n"
      "v_min_u32_dpp v9, v9, v9 quad_perm:[2,3,0,1] row_mask:0xf bank_mask:0xf\n"
      "buffer_load_dwordx4 v[120:123], v1, s[64:67], s62 offen\n"
      "v_mfma_f32_16x16x128_f8f6f4 v[52:55], a[232:239], v[248:255], v[52:55] cbsz:1\n"
      "v_readfirstlane_b32 s94, v9\n"
      "s_sub_u32 s94, s94, s76\n"
      "s_min_i32 s94, s94, 32\n"
      "s_cmp_ge_i32 s94, 4\n"
      "buffer_load_dwordx4 v[124:127], v1, s[64:67], s63 offen\n"
      "v_mfma_f32_16x16x128_f8f6f4 v[56:59], a[240:247], v[248:255], v[56:59] cbsz:1\n"
      "s_cselect_b32 s95, %[nown], 0\n"
      "s_add_i32 s94, s94, s95\n"
      "s_min_i32 s94, s94, s40\n"
      "s_max_i32 s74, s74, s94\n"
      "v_mfma_f32_16x16x128_f8f6f4 v[60:63], a[248:255], v[248:255], v[60:63] cbsz:1\n"
      "s_add_u32 s72, s72, 1\n"
      "s_cmp_le_u32 s73, s72\n"
      "s_cbranch_scc1 .Lslow0_%=\n"
      "s_branch .Lgo0_%=\n"
      ".Lslow0_%=:\n"
      "s_cmp_eq_u32 s72, %[nb]\n"
      "s_cbranch_scc1 .Lflush2_%=\n"
      "s_memrealtime s[80:81]\n"
      "s_mov_b32 s84, 0\n"
      "s_mov_b32 s66, 65536\n"
      "s_lshl_b32 s95, s43, 16\n"
      "s_add_u32 s64, s77, s95\n"
      "s_addc_u32 s65, s78, 0\n"
      "s_cmp_eq_u32 s46, 0\n"
      "s_cbranch_scc1 .Lnospec0_%=\n"
      "s_waitcnt vmcnt(9)\n"
      "v_min3_u32 v9, v64, v68, v72\n"
      "v_min3_u32 v9, v9, v76, v80\n"
      "v_min3_u32 v9, v9, v84, v88\n"
      "v_min_u32 v9, v9, v92\n"
      "v_cmp_eq_u32 vcc, 0, v9\n"
      "s_nop 4\n"
      "s_mov_b32 s41, 1\n"
      "s_cbranch_vccz .Lgot0_%=\n"
      "s_mov_b32 s41, 0\n"
      "s_branch .Lnospec0_%=\n"
      ".Lval0_%=:\n"
      "v_min3_u32 v9, v64, v68, v72\n"
      "v_min3_u32 v9, v9, v76, v80\n"
      "v_min3_u32 v9, v9, v84, v88\n"
      "v_min3_u32 v9, v9, v92, v96\n"
      "v_min3_u32 v9, v9, v100, v104\n"
      "v_min3_u32 v9, v9, v108, v112\n"
      "v_min3_u32 v9, v9, v116, v120\n"
      "v_min_u32 v9, v9, v124\n"
      "v_cmp_eq_u32 vcc, 0, v9\n"
      "s_nop 4\n"
      "s_cbranch_vccz .Lgot0_%=\n"
      ".Lnospec0_%=:\n"
      "s_cmp_gt_u32 s74, s72\n"
      "s_cbranch_scc1 .Lhave0_%=\n"
      "s_cmp_lt_u32 s72, s47\n"
      "s_cbranch_scc1 .Lpoll0_%=\n"
      "s_mov_b32 s38, 4096\n"
      "buffer_load_dword v8, v4, s[36:39], %[po] offen sc1\n"
      "buffer_load_dwordx4 v[64:67], v1, s[64:67], s48 offen sc1\n"
      "buffer_load_dwordx4 v[68:71], v1, s[64:67], s49 offen sc1\n"
      "buffer_load_dwordx4 v[72:75], v1, s[64:67], s50 offen sc1\n"
      "buffer_load_dwordx4 v[76:79], v1, s[64:67], s51 offen sc1\n"
      "buffer_load_dwordx4 v[80:83], v1, s[64:67], s52 offen sc1\n"
      "buffer_load_dwordx4 v[84:87], v1, s[64:67], s53 offen sc1\n"
      "buffer_load_dwordx4 v[88:91], v1, s[64:67], s54 offen sc1\n"
      "buffer_load_dwordx4 v[92:95], v1, s[64:67], s55 offen sc1\n"
      "buffer_load_dwordx4 v[96:99], v1, s[64:67], s56 offen sc1\n"
      "buffer_load_dwordx4 v[100:103], v1, s[64:67], s57 offen sc1\n"
      "buffer_load_dwordx4 v[104:107], v1, s[64:67], s58 offen sc1\n"
      "buffer_load_dwordx4 v[108:111], v1, s[64:67], s59 offen sc1\n"
      "buffer_load_dwordx4 v[112:115], v1, s[64:67], s60 offen sc1\n"
      "buffer_load_dwordx4 v[116:119], v1, s[64:67], s61 offen sc1\n"
      "buffer_load_dwordx4 v[120:123], v1, s[64:67], s62 offen sc1\n"
      "buffer_load_dwordx4 v[124:127], v1, s[64:67], s63 offen sc1\n"
      "s_waitcnt vmcnt(0)\n"
      "v_min_u32_dpp v9, v8, v8 quad_perm:[1,0,3,2] row_mask:0xf bank_mask:0xf\n"
      "s_nop 1\n"
      "v_min_u32_dpp v9, v9, v9 quad_perm:[2,3,0,1] row_mask:0xf bank_mask:0xf\n"
      "s_nop 0\n"
      "v_readfirstlane_b32 s94, v9\n"
      "s_sub_u32 s94, s94, s76\n"
      "s_min_i32 s94, s94, 32\n"
      "s_cmp_ge_i32 s94, 4\n"
      "s_cselect_b32 s95, %[nown], 0\n"
      "s_add_i32 s94, s94, s95\n"
      "s_min_i32 s94, s94, s40\n"
      "s_max_i32 s74, s74, s94\n"
      "s_add_u32 s84, s84, 1\n"
      "s_and_b32 s94, s84, 63\n"
      "s_cbranch_scc1 .Lval0_%=\n"
      "v_mov_b32 v9, 8192\n"
      "global_load_dword v9, v9, s[98:99] sc1\n"
      "s_memrealtime s[82:83]\n"
      "s_waitcnt vmcnt(0) lgkmcnt(0)\n"
      "v_readfirstlane_b32 s94, v9\n"
      "s_sub_u32 s95, s82, s80\n"
      "s_cmp_eq_u32 s94, s75\n"
      "s_cbranch_scc1 .Ldead0_%=\n"
      "s_cmp_lt_u32 s95, 30000000\n"
      "s_cbranch_scc1 .Lval0_%=\n"
      "s_branch .Ldead0_%=\n"
      ".Lpoll0_%=:\n"
      "s_mov_b32 s38, 4096\n"
      "buffer_load_dword v8, v4, s[36:39], %[po] offen sc1\n"
      "s_waitcnt vmcnt(0)\n"
      "v_min_u32_dpp v9, v8, v8 quad_perm:[1,0,3,2] row_mask:0xf bank_mask:0xf\n"
      "s_nop 1\n"
      "v_min_u32_dpp v9, v9, v9 quad_perm:[2,3,0,1] row_mask:0xf bank_mask:0xf\n"
      "s_nop 0\n"
      "v_readfirstlane_b32 s94, v9\n"
      "s_sub_u32 s94, s94, s76\n"
      "s_min_i32 s94, s94, 32\n"
      "s_cmp_ge_i32 s94, 4\n"
      "s_cselect_b32 s95, %[nown], 0\n"
      "s_add_i32 s94, s94, s95\n"
      "s_min_i32 s94, s94, s40\n"
      "s_max_i32 s74, s74, s94\n"
      "s_cmp_gt_u32 s74, s72\n"
      "s_cbranch_scc1 .Lhave0_%=\n"
      "s_add_u32 s84, s84, 1\n"
      "s_and_b32 s94, s84, 63\n"
      "s_cbranch_scc1 .Lpoll0_%=\n"
      "v_mov_b32 v9, 8192\n"
      "global_load_dword v9, v9, s[98:99] sc1\n"
      "s_memrealtime s[82:83]\n"
      "s_waitcnt vmcnt(0) lgkmcnt(0)\n"
      "v_readfirstlane_b32 s94, v9\n"
      "s_sub_u32 s95, s82, s80\n"
      "s_cmp_eq_u32 s94, s75\n"
      "s_cbranch_scc1 .Ldead0_%=\n"
      "s_cmp_lt_u32 s95, 30000000\n"
      "s_cbranch_scc1 .Lpoll0_%=\n"
      "s_branch .Ldead0_%=\n"
      ".Ldead0_%=:\n"
      "v_mov_b32 v9, 8192\n"
      "v_mov_b32 v8, s75\n"
      "global_store_dword v9, v8, s[98:99] sc1\n"
      "s_mov_b32 s79, 0x7fc00000\n"
      "s_mov_b32 s74, %[nb]\n"
      ".Lhave0_%=:\n"
      "buffer_load_dwordx4 v[64:67], v1, s[64:67], s48 offen sc1\n"
      "buffer_load_dwordx4 v[68:71], v1, s[64:67], s49 offen sc1\n"
      "buffer_load_dwordx4 v[72:75], v1, s[64:67], s50 offen sc1\n"
      "buffer_load_dwordx4 v[76:79], v1, s[64:67], s51 offen sc1\n"
      "buffer_load_dwordx4 v[80:83], v1, s[64:67], s52 offen sc1\n"
      "buffer_load_dwordx4 v[84:87], v1, s[64:67], s53 offen sc1\n"
      "buffer_load_dwordx4 v[88:91], v1, s[64:67], s54 offen sc1\n"
      "buffer_load_dwordx4 v[92:95], v1, s[64:67], s55 offen sc1\n"
      "buffer_load_dwordx4 v[96:99], v1, s[64:67], s56 offen sc1\n"
      "buffer_load_dwordx4 v[100:103], v1, s[64:67], s57 offen sc1\n"
      "buffer_load_dwordx4 v[104:107], v1, s[64:67], s58 offen sc1\n"
      "buffer_load_dwordx4 v[108:111], v1, s[64:67], s59 offen sc1\n"
      "buffer_load_dwordx4 v[112:115], v1, s[64:67], s60 offen sc1\n"
      "buffer_load_dwordx4 v[116:119], v1, s[64:67], s61 offen sc1\n"
      "buffer_load_dwordx4 v[120:123], v1, s[64:67], s62 offen sc1\n"
      "buffer_load_dwordx4 v[124:127], v1, s[64:67], s63 offen sc1\n"
      ".Lgot0_%=:\n"
      "s_add_u32 s73, s72, 1\n"
      "s_branch .Lgo0_%=\n"
      ".Lslow1_%=:\n"
      "s_cmp_eq_u32 s72, %[nb]\n"
      "s_cbranch_scc1 .Lflush0_%=\n"
      "s_memrealtime s[80:81]\n"
      "s_mov_b32 s84, 0\n"
      "s_cmp_lg_u32 s72, 4\n"
      "s_cbranch_scc1 .Lnoown1_%=\n"
      ".Lown1_%=:\n"
      "s_mov_b32 s38, 4096\n"
      "buffer_load_dword v8, v4, s[36:39], %[opo] offen sc1\n"
      "s_waitcnt vmcnt(0)\n"
      "v_min_u32_dpp v9, v8, v8 quad_perm:[1,0,3,2] row_mask:0xf bank_mask:0xf\n"
      "s_nop 1\n"
      "v_min_u32_dpp v9, v9, v9 quad_perm:[2,3,0,1] row_mask:0xf bank_mask:0xf\n"
      "s_nop 0\n"
      "v_readfirstlane_b32 s94, v9\n"
      "s_cmp_ge_u32 s94, s75\n"
      "s_cbranch_scc1 .Lownok1_%=\n"
      "s_add_u32 s84, s84, 1\n"
      "s_and_b32 s94, s84, 63\n"
      "s_cbranch_scc1 .Lown1_%=\n"
      "v_mov_b32 v9, 8192\n"
      "global_load_dword v9, v9, s[98:99] sc1\n"
      "s_memrealtime s[82:83]\n"
      "s_waitcnt vmcnt(0) lgkmcnt(0)\n"
      "v_readfirstlane_b32 s94, v9\n"
      "s_sub_u32 s95, s82, s80\n"
      "s_cmp_eq_u32 s94, s75\n"
      "s_cbranch_scc1 .Lowndead1_%=\n"
      "s_cmp_lt_u32 s95, 30000000\n"
      "s_cbranch_scc1 .Lown1_%=\n"
      "s_branch .Lowndead1_%=\n"
      ".Lowndead1_%=:\n"
      "v_mov_b32 v9, 8192\n"
      "v_mov_b32 v8, s75\n"
      "global_store_dword v9, v8, s[98:99] sc1\n"
      "s_mov_b32 s79, 0x7fc00000\n"
      "s_mov_b32 s74, %[nb]\n"
      ".Lownok1_%=:\n"
      "s_mov_b32 s40, 64\n"
      "s_max_u32 s74, s74, %[nownr]\n"
      "v_mov_b32 v8, 0\n"
      ".Lnoown1_%=:\n"
      "s_mov_b32 s66, 65536\n"
      "s_lshl_b32 s95, s43, 16\n"
      "s_add_u32 s64, s77, s95\n"
      "s_addc_u32 s65, s78, 0\n"
      "s_cmp_eq_u32 s46, 0\n"
      "s_cbranch_scc1 .Lnospec1_%=\n"
      "s_waitcnt vmcnt(9)\n"
      "v_min3_u32 v9, v128, v132, v136\n"
      "v_min3_u32 v9, v9, v140, v144\n"
      "v_min3_u32 v9, v9, v148, v152\n"
      "v_min_u32 v9, v9, v156\n"
      "v_cmp_eq_u32 vcc, 0, v9\n"
      "s_nop 4\n"
      "s_mov_b32 s41, 1\n"
      "s_cbranch_vccz .Lgot1_%=\n"
      "s_mov_b32 s41, 0\n"
      "s_branch .Lnospec1_%=\n"
      ".Lval1_%=:\n"
      "v_min3_u32 v9, v128, v132, v136\n"
      "v_min3_u32 v9, v9, v140, v144\n"
      "v_min3_u32 v9, v9, v148, v152\n"
      "v_min3_u32 v9, v9, v156, v160\n"
      "v_min3_u32 v9, v9, v164, v168\n"
      "v_min3_u32 v9, v9, v172, v176\n"
      "v_min3_u32 v9, v9, v180, v184\n"
      "v_min_u32 v9, v9, v188\n"
      "v_cmp_eq_u32 vcc, 0, v9\n"
      "s_nop 4\n"
      "s_cbranch_vccz .Lgot1_%=\n"
      ".Lnospec1_%=:\n"
      "s_cmp_gt_u32 s74, s72\n"
      "s_cbranch_scc1 .Lhave1_%=\n"
      "s_cmp_lt_u32 s72, s47\n"
      "s_cbranch_scc1 .Lpoll1_%=\n"
      "s_mov_b32 s38, 4096\n"
      "buffer_load_dword v8, v4, s[36:39], %[po] offen sc1\n"
      "buffer_load_dwordx4 v[128:131], v1, s[64:67], s48 offen sc1\n"
      "buffer_load_dwordx4 v[132:135], v1, s[64:67], s49 offen sc1\n"
      "buffer_load_dwordx4 v[136:139], v1, s[64:67], s50 offen sc1\n"
      "buffer_load_dwordx4 v[140:143], v1, s[64:67], s51 offen sc1\n"
      "buffer_load_dwordx4 v[144:147], v1, s[64:67], s52 offen sc1\n"
      "buffer_load_dwordx4 v[148:151], v1, s[64:67], s53 offen sc1\n"
      "buffer_load_dwordx4 v[152:155], v1, s[64:67], s54 offen sc1\n"
      "buffer_load_dwordx4 v[156:159], v1, s[64:67], s55 offen sc1\n"
      "buffer_load_dwordx4 v[160:163], v1, s[64:67], s56 offen sc1\n"
      "buffer_load_dwordx4 v[164:167], v1, s[64:67], s57 offen sc1\n"
      "buffer_load_dwordx4 v[168:171], v1, s[64:67], s58 offen sc1\n"
      "buffer_load_dwordx4 v[172:175], v1, s[64:67], s59 offen sc1\n"
      "buffer_load_dwordx4 v[176:179], v1, s[64:67], s60 offen sc1\n"
      "buffer_load_dwordx4 v[180:183], v1, s[64:67], s61 offen sc1\n"
      "buffer_load_dwordx4 v[184:187], v1, s[64:67], s62 offen sc1\n"
      "buffer_load_dwordx4 v[188:191], v1, s[64:67], s63 offen sc1\n"
      "s_waitcnt vmcnt(0)\n"
      "v_min_u32_dpp v9, v8, v8 quad_perm:[1,0,3,2] row_mask:0xf bank_mask:0xf\n"
      "s_nop 1\n"
      "v_min_u32_dpp v9, v9, v9 quad_perm:[2,3,0,1] row_mask:0xf bank_mask:0xf\n"
      "s_nop 0\n"
      "v_readfirstlane_b32 s94, v9\n"
      "s_sub_u32 s94, s94, s76\n"
      "s_min_i32 s94, s94, 32\n"
      "s_cmp_ge_i32 s94, 4\n"
      "s_cselect_b32 s95, %[nown], 0\n"
      "s_add_i32 s94, s94, s95\n"
      "s_min_i32 s94, s94, s40\n"
      "s_max_i32 s74, s74, s94\n"
      "s_add_u32 s84, s84, 1\n"
      "s_and_b32 s94, s84, 63\n"
      "s_cbranch_scc1 .Lval1_%=\n"
      "v_mov_b32 v9, 8192\n"
      "global_load_dword v9, v9, s[98:99] sc1\n"
      "s_memrealtime s[82:83]\n"
      "s_waitcnt vmcnt(0) lgkmcnt(0)\n"
      "v_readfirstlane_b32 s94, v9\n"
      "s_sub_u32 s95, s82, s80\n"
      "s_cmp_eq_u32 s94, s75\n"
      "s_cbranch_scc1 .Ldead1_%=\n"
      "s_cmp_lt_u32 s95, 30000000\n"
      "s_cbranch_scc1 .Lval1_%=\n"
      "s_branch .Ldead1_%=\n"
      ".Lpoll1_%=:\n"
      "s_mov_b32 s38, 4096\n"
      "buffer_load_dword v8, v4, s[36:39], %[po] offen sc1\n"
      "s_waitcnt vmcnt(0)\n"
      "v_min_u32_dpp v9, v8, v8 quad_perm:[1,0,3,2] row_mask:0xf bank_mask:0xf\n"
      "s_nop 1\n"
      "v_min_u32_dpp v9, v9, v9 quad_perm:[2,3,0,1] row_mask:0xf bank_mask:0xf\n"
      "s_nop 0\n"
      "v_readfirstlane_b32 s94, v9\n"
      "s_sub_u32 s94, s94, s76\n"
      "s_min_i32 s94, s94, 32\n"
      "s_cmp_ge_i32 s94, 4\n"
      "s_cselect_b32 s95, %[nown], 0\n"
      "s_add_i32 s94, s94, s95\n"
      "s_min_i32 s94, s94, s40\n"
      "s_max_i32 s74, s74, s94\n"
      "s_cmp_gt_u32 s74, s72\n"
      "s_cbranch_scc1 .Lhave1_%=\n"
      "s_add_u32 s84, s84, 1\n"
      "s_and_b32 s94, s84, 63\n"
      "s_cbranch_scc1 .Lpoll1_%=\n"
      "v_mov_b32 v9, 8192\n"
      "global_load_dword v9, v9, s[98:99] sc1\n"
      "s_memrealtime s[82:83]\n"
      "s_waitcnt vmcnt(0) lgkmcnt(0)\n"
      "v_readfirstlane_b32 s94, v9\n"
      "s_sub_u32 s95, s82, s80\n"
      "s_cmp_eq_u32 s94, s75\n"
      "s_cbranch_scc1 .Ldead1_%=\n"
      "s_cmp_lt_u32 s95, 30000000\n"
      "s_cbranch_scc1 .Lpoll1_%=\n"
      "s_branch .Ldead1_%=\n"
      ".Ldead1_%=:\n"
      "v_mov_b32 v9, 8192\n"
      "v_mov_b32 v8, s75\n"
      "global_store_dword v9, v8, s[98:99] sc1\n"
      "s_mov_b32 s79, 0x7fc00000\n"
      "s_mov_b32 s74, %[nb]\n"
      ".Lhave1_%=:\n"
      "buffer_load_dwordx4 v[128:131], v1, s[64:67], s48 offen sc1\n"
      "buffer_load_dwordx4 v[132:135], v1, s[64:67], s49 offen sc1\n"
      "buffer_load_dwordx4 v[136:139], v1, s[64:67], s50 offen sc1\n"
      "buffer_load_dwordx4 v[140:143], v1, s[64:67], s51 offen sc1\n"
      "buffer_load_dwordx4 v[144:147], v1, s[64:67], s52 offen sc1\n"
      "buffer_load_dwordx4 v[148:151], v1, s[64:67], s53 offen sc1\n"
      "buffer_load_dwordx4 v[152:155], v1, s[64:67], s54 offen sc1\n"
      "buffer_load_dwordx4 v[156:159], v1, s[64:67], s55 offen sc1\n"
      "buffer_load_dwordx4 v[160:163], v1, s[64:67], s56 offen sc1\n"
      "buffer_load_dwordx4 v[164:167], v1, s[64:67], s57 offen sc1\n"
      "buffer_load_dwordx4 v[168:171], v1, s[64:67], s58 offen sc1\n"
      "buffer_load_dwordx4 v[172:175], v1, s[64:67], s59 offen sc1\n"
      "buffer_load_dwordx4 v[176:179], v1, s[64:67], s60 offen sc1\n"
      "buffer_load_dwordx4 v[180:183], v1, s[64:67], s61 offen sc1\n"
      "buffer_load_dwordx4 v[184:187], v1, s[64:67], s62 offen sc1\n"
      "buffer_load_dwordx4 v[188:191], v1, s[64:67], s63 offen sc1\n"
      ".Lgot1_%=:\n"
      "s_add_u32 s73, s72, 1\n"
      "s_branch .Lgo1_%=\n"
      ".Lslow2_%=:\n"
      "s_cmp_eq_u32 s72, %[nb]\n"
      "s_cbranch_scc1 .Lflush1_%=\n"
      "s_memrealtime s[80:81]\n"
      "s_mov_b32 s84, 0\n"
      "s_mov_b32 s66, 65536\n"
      "s_lshl_b32 s95, s43, 16\n"
      "s_add_u32 s64, s77, s95\n"
      "s_addc_u32 s65, s78, 0\n"
      "s_cmp_eq_u32 s46, 0\n"
      "s_cbranch_scc1 .Lnospec2_%=\n"
      "s_waitcnt vmcnt(9)\n"
      "v_min3_u32 v9, v192, v196, v200\n"
      "v_min3_u32 v9, v9, v204, v208\n"
      "v_min3_u32 v9, v9, v212, v216\n"
      "v_min_u32 v9, v9, v220\n"
      "v_cmp_eq_u32 vcc, 0, v9\n"
      "s_nop 4\n"
      "s_mov_b32 s41, 1\n"
      "s_cbranch_vccz .Lgot2_%=\n"
      "s_mov_b32 s41, 0\n"
      "s_branch .Lnospec2_%=\n"
      ".Lval2_%=:\n"
      "v_min3_u32 v9, v192, v196, v200\n"
      "v_min3_u32 v9, v9, v204, v208\n"
      "v_min3_u32 v9, v9, v212, v216\n"
      "v_min3_u32 v9, v9, v220, v224\n"
      "v_min3_u32 v9, v9, v228, v232\n"
      "v_min3_u32 v9, v9, v236, v240\n"
      "v_min3_u32 v9, v9, v244, v248\n"
      "v_min_u32 v9, v9, v252\n"
      "v_cmp_eq_u32 vcc, 0, v9\n"
      "s_nop 4\n"
      "s_cbranch_vccz .Lgot2_%=\n"
      ".Lnospec2_%=:\n"
      "s_cmp_gt_u32 s74, s72\n"
      "s_cbranch_scc1 .Lhave2_%=\n"
      "s_cmp_lt_u32 s72, s47\n"
      "s_cbranch_scc1 .Lpoll2_%=\n"
      "s_mov_b32 s38, 4096\n"
      "buffer_load_dword v8, v4, s[36:39], %[po] offen sc1\n"
      "buffer_load_dwordx4 v[192:195], v1, s[64:67], s48 offen sc1\n"
      "buffer_load_dwordx4 v[196:199], v1, s[64:67], s49 offen sc1\n"
      "buffer_load_dwordx4 v[200:203], v1, s[64:67], s50 offen sc1\n"
      "buffer_load_dwordx4 v[204:207], v1, s[64:67], s51 offen sc1\n"
      "buffer_load_dwordx4 v[208:211], v1, s[64:67], s52 offen sc1\n"
      "buffer_load_dwordx4 v[212:215], v1, s[64:67], s53 offen sc1\n"
      "buffer_load_dwordx4 v[216:219], v1, s[64:67], s54 offen sc1\n"
      "buffer_load_dwordx4 v[220:223], v1, s[64:67], s55 offen sc1\n"
      "buffer_load_dwordx4 v[224:227], v1, s[64:67], s56 offen sc1\n"
      "buffer_load_dwordx4 v[228:231], v1, s[64:67], s57 offen sc1\n"
      "buffer_load_dwordx4 v[232:235], v1, s[64:67], s58 offen sc1\n"
      "buffer_load_dwordx4 v[236:239], v1, s[64:67], s59 offen sc1\n"
      "buffer_load_dwordx4 v[240:243], v1, s[64:67], s60 offen sc1\n"
      "buffer_load_dwordx4 v[244:247], v1, s[64:67], s61 offen sc1\n"
      "buffer_load_dwordx4 v[248:251], v1, s[64:67], s62 offen sc1\n"
      "buffer_load_dwordx4 v[252:255], v1, s[64:67], s63 offen sc1\n"
      "s_waitcnt vmcnt(0)\n"
      "v_min_u32_dpp v9, v8, v8 quad_perm:[1,0,3,2] row_mask:0xf bank_mask:0xf\n"
      "s_nop 1\n"
      "v_min_u32_dpp v9, v9, v9 quad_perm:[2,3,0,1] row_mask:0xf bank_mask:0xf\n"
      "s_nop 0\n"
      "v_readfirstlane_b32 s94, v9\n"
      "s_sub_u32 s94, s94, s76\n"
      "s_min_i32 s94, s94, 32\n"
      "s_cmp_ge_i32 s94, 4\n"
      "s_cselect_b32 s95, %[nown], 0\n"
      "s_add_i32 s94, s94, s95\n"
      "s_min_i32 s94, s94, s40\n"
      "s_max_i32 s74, s74, s94\n"
      "s_add_u32 s84, s84, 1\n"
      "s_and_b32 s94, s84, 63\n"
      "s_cbranch_scc1 .Lval2_%=\n"
      "v_mov_b32 v9, 8192\n"
      "global_load_dword v9, v9, s[98:99] sc1\n"
      "s_memrealtime s[82:83]\n"
      "s_waitcnt vmcnt(0) lgkmcnt(0)\n"
      "v_readfirstlane_b32 s94, v9\n"
      "s_sub_u32 s95, s82, s80\n"
      "s_cmp_eq_u32 s94, s75\n"
      "s_cbranch_scc1 .Ldead2_%=\n"
      "s_cmp_lt_u32 s95, 30000000\n"
      "s_cbranch_scc1 .Lval2_%=\n"
      "s_branch .Ldead2_%=\n"
      ".Lpoll2_%=:\n"
      "s_mov_b32 s38, 4096\n"
      "buffer_load_dword v8, v4, s[36:39], %[po] offen sc1\n"
      "s_waitcnt vmcnt(0)\n"
      "v_min_u32_dpp v9, v8, v8 quad_perm:[1,0,3,2] row_mask:0xf bank_mask:0xf\n"
      "s_nop 1\n"
      "v_min_u32_dpp v9, v9, v9 quad_perm:[2,3,0,1] row_mask:0xf bank_mask:0xf\n"
      "s_nop 0\n"
      "v_readfirstlane_b32 s94, v9\n"
      "s_sub_u32 s94, s94, s76\n"
      "s_min_i32 s94, s94, 32\n"
      "s_cmp_ge_i32 s94, 4\n"
      "s_cselect_b32 s95, %[nown], 0\n"
      "s_add_i32 s94, s94, s95\n"
      "s_min_i32 s94, s94, s40\n"
      "s_max_i32 s74, s74, s94\n"
      "s_cmp_gt_u32 s74, s72\n"
      "s_cbranch_scc1 .Lhave2_%=\n"
      "s_add_u32 s84, s84, 1\n"
      "s_and_b32 s94, s84, 63\n"
      "s_cbranch_scc1 .Lpoll2_%=\n"
      "v_mov_b32 v9, 8192\n"
      "global_load_dword v9, v9, s[98:99] sc1\n"
      "s_memrealtime s[82:83]\n"
      "s_waitcnt vmcnt(0) lgkmcnt(0)\n"
      "v_readfirstlane_b32 s94, v9\n"
      "s_sub_u32 s95, s82, s80\n"
      "s_cmp_eq_u32 s94, s75\n"
      "s_cbranch_scc1 .Ldead2_%=\n"
      "s_cmp_lt_u32 s95, 30000000\n"
      "s_cbranch_scc1 .Lpoll2_%=\n"
      "s_branch .Ldead2_%=\n"
      ".Ldead2_%=:\n"
      "v_mov_b32 v9, 8192\n"
      "v_mov_b32 v8, s75\n"
      "global_store_dword v9, v8, s[98:99] sc1\n"
      "s_mov_b32 s79, 0x7fc00000\n"
      "s_mov_b32 s74, %[nb]\n"
      ".Lhave2_%=:\n"
      "buffer_load_dwordx4 v[192:195], v1, s[64:67], s48 offen sc1\n"
      "buffer_load_dwordx4 v[196:199], v1, s[64:67], s49 offen sc1\n"
      "buffer_load_dwordx4 v[200:203], v1, s[64:67], s50 offen sc1\n"
      "buffer_load_dwordx4 v[204:207], v1, s[64:67], s51 offen sc1\n"
      "buffer_load_dwordx4 v[208:211], v1, s[64:67], s52 offen sc1\n"
      "buffer_load_dwordx4 v[212:215], v1, s[64:67], s53 offen sc1\n"
      "buffer_load_dwordx4 v[216:219], v1, s[64:67], s54 offen sc1\n"
      "buffer_load_dwordx4 v[220:223], v1, s[64:67], s55 offen sc1\n"
      "buffer_load_dwordx4 v[224:227], v1, s[64:67], s56 offen sc1\n"
      "buffer_load_dwordx4 v[228:231], v1, s[64:67], s57 offen sc1\n"
      "buffer_load_dwordx4 v[232:235], v1, s[64:67], s58 offen sc1\n"
      "buffer_load_dwordx4 v[236:239], v1, s[64:67], s59 offen sc1\n"
      "buffer_load_dwordx4 v[240:243], v1, s[64:67], s60 offen sc1\n"
      "buffer_load_dwordx4 v[244:247], v1, s[64:67], s61 offen sc1\n"
      "buffer_load_dwordx4 v[248:251], v1, s[64:67], s62 offen sc1\n"
      "buffer_load_dwordx4 v[252:255], v1, s[64:67], s63 offen sc1\n"
      ".Lgot2_%=:\n"
      "s_add_u32 s73, s72, 1\n"
      "s_branch .Lgo2_%=\n"
      ".Lmid0_%=:\n"
      "s_mov_b32 s41, 0\n"
      "s_waitcnt vmcnt(9)\n"
      "v_min3_u32 v9, v96, v100, v104\n"
      "v_min3_u32 v9, v9, v108, v112\n"
      "v_min3_u32 v9, v9, v116, v120\n"
      "v_min_u32 v9, v9, v124\n"
      "v_cmp_eq_u32 vcc, 0, v9\n"
      "s_nop 4\n"
      "s_cbranch_vccz .Lmidret0_%=\n"
      "s_mov_b64 s[86:87], s[64:65]\n"
      "s_mov_b32 s88, s66\n"
      "s_mov_b32 s66, 65536\n"
      "s_lshl_b32 s95, s42, 16\n"
      "s_add_u32 s64, s77, s95\n"
      "s_addc_u32 s65, s78, 0\n"
      "s_memrealtime s[80:81]\n"
      "s_mov_b32 s84, 0\n"
      ".Lmidre0_%=:\n"
      "s_mov_b32 s89, s74\n"
      "s_mov_b32 s38, 4096\n"
      "buffer_load_dword v8, v4, s[36:39], %[po] offen sc1\n"
      "buffer_load_dwordx4 v[96:99], v1, s[64:67], s56 offen sc1\n"
      "buffer_load_dwordx4 v[100:103], v1, s[64:67], s57 offen sc1\n"
      "buffer_load_dwordx4 v[104:107], v1, s[64:67], s58 offen sc1\n"
      "buffer_load_dwordx4 v[108:111], v1, s[64:67], s59 offen sc1\n"
      "buffer_load_dwordx4 v[112:115], v1, s[64:67], s60 offen sc1\n"
      "buffer_load_dwordx4 v[116:119], v1, s[64:67], s61 offen sc1\n"
      "buffer_load_dwordx4 v[120:123], v1, s[64:67], s62 offen sc1\n"
      "buffer_load_dwordx4 v[124:127], v1, s[64:67], s63 offen sc1\n"
      "s_waitcnt vmcnt(0)\n"
      "s_cmp_gt_u32 s89, s72\n"
      "s_cbranch_scc1 .Lmidok0_%=\n"
      "v_min_u32_dpp v9, v8, v8 quad_perm:[1,0,3,2] row_mask:0xf bank_mask:0xf\n"
      "s_nop 1\n"
      "v_min_u32_dpp v9, v9, v9 quad_perm:[2,3,0,1] row_mask:0xf bank_mask:0xf\n"
      "s_nop 0\n"
      "v_readfirstlane_b32 s94, v9\n"
      "s_sub_u32 s94, s94, s76\n"
      "s_min_i32 s94, s94, 32\n"
      "s_cmp_ge_i32 s94, 4\n"
      "s_cselect_b32 s95, %[nown], 0\n"
      "s_add_i32 s94, s94, s95\n"
      "s_min_i32 s94, s94, s40\n"
      "s_max_i32 s74, s74, s94\n"
      "v_min3_u32 v9, v96, v100, v104\n"
      "v_min3_u32 v9, v9, v108, v112\n"
      "v_min3_u32 v9, v9, v116, v120\n"
      "v_min_u32 v9, v9, v124\n"
      "v_cmp_eq_u32 vcc, 0, v9\n"
      "s_nop 4\n"
      "s_cbranch_vccz .Lmidok0_%=\n"
      "s_add_u32 s84, s84, 1\n"
      "s_and_b32 s94, s84, 63\n"
      "s_cbranch_scc1 .Lmidre0_%=\n"
      "v_mov_b32 v9, 8192\n"
      "global_load_dword v9, v9, s[98:99] sc1\n"
      "s_memrealtime s[82:83]\n"
      "s_waitcnt vmcnt(0) lgkmcnt(0)\n"
      "v_readfirstlane_b32 s94, v9\n"
      "s_sub_u32 s95, s82, s80\n"
      "s_cmp_eq_u32 s94, s75\n"
      "s_cbranch_scc1 .Lmiddead0_%=\n"
      "s_cmp_lt_u32 s95, 30000000\n"
      "s_cbranch_scc1 .Lmidre0_%=\n"
      "s_branch .Lmiddead0_%=\n"
      ".Lmiddead0_%=:\n"
      "v_mov_b32 v9, 8192\n"
      "v_mov_b32 v8, s75\n"
      "global_store_dword v9, v8, s[98:99] sc1\n"
      "s_mov_b32 s79, 0x7fc00000\n"
      "s_mov_b32 s74, %[nb]\n"
      ".Lmidok0_%=:\n"
      "s_mov_b64 s[64:65], s[86:87]\n"
      "s_mov_b32 s66, s88\n"
      "s_branch .Lmidret0_%=\n"
      ".Lmid1_%=:\n"
      "s_mov_b32 s41, 0\n"
      "s_waitcnt vmcnt(9)\n"
      "v_min3_u32 v9, v160, v164, v168\n"
      "v_min3_u32 v9, v9, v172, v176\n"
      "v_min3_u32 v9, v9, v180, v184\n"
      "v_min_u32 v9, v9, v188\n"
      "v_cmp_eq_u32 vcc, 0, v9\n"
      "s_nop 4\n"
      "s_cbranch_vccz .Lmidret1_%=\n"
      "s_mov_b64 s[86:87], s[64:65]\n"
      "s_mov_b32 s88, s66\n"
      "s_mov_b32 s66, 65536\n"
      "s_lshl_b32 s95, s42, 16\n"
      "s_add_u32 s64, s77, s95\n"
      "s_addc_u32 s65, s78, 0\n"
      "s_memrealtime s[80:81]\n"
      "s_mov_b32 s84, 0\n"
      ".Lmidre1_%=:\n"
      "s_mov_b32 s89, s74\n"
      "s_mov_b32 s38, 4096\n"
      "buffer_load_dword v8, v4, s[36:39], %[po] offen sc1\n"
      "buffer_load_dwordx4 v[160:163], v1, s[64:67], s56 offen sc1\n"
      "buffer_load_dwordx4 v[164:167], v1, s[64:67], s57 offen sc1\n"
      "buffer_load_dwordx4 v[168:171], v1, s[64:67], s58 offen sc1\n"
      "buffer_load_dwordx4 v[172:175], v1, s[64:67], s59 offen sc1\n"
      "buffer_load_dwordx4 v[176:179], v1, s[64:67], s60 offen sc1\n"
      "buffer_load_dwordx4 v[180:183], v1, s[64:67], s61 offen sc1\n"
      "buffer_load_dwordx4 v[184:187], v1, s[64:67], s62 offen sc1\n"
      "buffer_load_dwordx4 v[188:191], v1, s[64:67], s63 offen sc1\n"
      "s_waitcnt vmcnt(0)\n"
      "s_cmp_gt_u32 s89, s72\n"
      "s_cbranch_scc1 .Lmidok1_%=\n"
      "v_min_u32_dpp v9, v8, v8 quad_perm:[1,0,3,2] row_mask:0xf bank_mask:0xf\n"
      "s_nop 1\n"
      "v_min_u32_dpp v9, v9, v9 quad_perm:[2,3,0,1] row_mask:0xf bank_mask:0xf\n"
      "s_nop 0\n"
      "v_readfirstlane_b32 s94, v9\n"
      "s_sub_u32 s94, s94, s76\n"
      "s_min_i32 s94, s94, 32\n"
      "s_cmp_ge_i32 s94, 4\n"
      "s_cselect_b32 s95, %[nown], 0\n"
      "s_add_i32 s94, s94, s95\n"
      "s_min_i32 s94, s94, s40\n"
      "s_max_i32 s74, s74, s94\n"
      "v_min3_u32 v9, v160, v164, v168\n"
      "v_min3_u32 v9, v9, v172, v176\n"
      "v_min3_u32 v9, v9, v180, v184\n"
      "v_min_u32 v9, v9, v188\n"
      "v_cmp_eq_u32 vcc, 0, v9\n"
      "s_nop 4\n"
      "s_cbranch_vccz .Lmidok1_%=\n"
      "s_add_u32 s84, s84, 1\n"
      "s_and_b32 s94, s84, 63\n"
      "s_cbranch_scc1 .Lmidre1_%=\n"
      "v_mov_b32 v9, 8192\n"
      "global_load_dword v9, v9, s[98:99] sc1\n"
      "s_memrealtime s[82:83]\n"
      "s_waitcnt vmcnt(0) lgkmcnt(0)\n"
      "v_readfirstlane_b32 s94, v9\n"
      "s_sub_u32 s95, s82, s80\n"
      "s_cmp_eq_u32 s94, s75\n"
      "s_cbranch_scc1 .Lmiddead1_%=\n"
      "s_cmp_lt_u32 s95, 30000000\n"
      "s_cbranch_scc1 .Lmidre1_%=\n"
      "s_branch .Lmiddead1_%=\n"
      ".Lmiddead1_%=:\n"
      "v_mov_b32 v9, 8192\n"
      "v_mov_b32 v8, s75\n"
      "global_store_dword v9, v8, s[98:99] sc1\n"
      "s_mov_b32 s79, 0x7fc00000\n"
      "s_mov_b32 s74, %[nb]\n"
      ".Lmidok1_%=:\n"
      "s_mov_b64 s[64:65], s[86:87]\n"
      "s_mov_b32 s66, s88\n"
      "s_branch .Lmidret1_%=\n"
      ".Lmid2_%=:\n"
      "s_mov_b32 s41, 0\n"
      "s_waitcnt vmcnt(9)\n"
      "v_min3_u32 v9, v224, v228, v232\n"
      "v_min3_u32 v9, v9, v236, v240\n"
      "v_min3_u32 v9, v9, v244, v248\n"
      "v_min_u32 v9, v9, v252\n"
      "v_cmp_eq_u32 vcc, 0, v9\n"
      "s_nop 4\n"
      "s_cbranch_vccz .Lmidret2_%=\n"
      "s_mov_b64 s[86:87], s[64:65]\n"
      "s_mov_b32 s88, s66\n"
      "s_mov_b32 s66, 65536\n"
      "s_lshl_b32 s95, s42, 16\n"
      "s_add_u32 s64, s77, s95\n"
      "s_addc_u32 s65, s78, 0\n"
      "s_memrealtime s[80:81]\n"
      "s_mov_b32 s84, 0\n"
      ".Lmidre2_%=:\n"
      "s_mov_b32 s89, s74\n"
      "s_mov_b32 s38, 4096\n"
      "buffer_load_dword v8, v4, s[36:39], %[po] offen sc1\n"
      "buffer_load_dwordx4 v[224:227], v1, s[64:67], s56 offen sc1\n"
      "buffer_load_dwordx4 v[228:231], v1, s[64:67], s57 offen sc1\n"
      "buffer_load_dwordx4 v[232:235], v1, s[64:67], s58 offen sc1\n"
      "buffer_load_dwordx4 v[236:239], v1, s[64:67], s59 offen sc1\n"
      "buffer_load_dwordx4 v[240:243], v1, s[64:67], s60 offen sc1\n"
      "buffer_load_dwordx4 v[244:247], v1, s[64:67], s61 offen sc1\n"
      "buffer_load_dwordx4 v[248:251], v1, s[64:67], s62 offen sc1\n"
      "buffer_load_dwordx4 v[252:255], v1, s[64:67], s63 offen sc1\n"
      "s_waitcnt vmcnt(0)\n"
      "s_cmp_gt_u32 s89, s72\n"
      "s_cbranch_scc1 .Lmidok2_%=\n"
      "v_min_u32_dpp v9, v8, v8 quad_perm:[1,0,3,2] row_mask:0xf bank_mask:0xf\n"
      "s_nop 1\n"
      "v_min_u32_dpp v9, v9, v9 quad_perm:[2,3,0,1] row_mask:0xf bank_mask:0xf\n"
      "s_nop 0\n"
      "v_readfirstlane_b32 s94, v9\n"
      "s_sub_u32 s94, s94, s76\n"
      "s_min_i32 s94, s94, 32\n"
      "s_cmp_ge_i32 s94, 4\n"
      "s_cselect_b32 s95, %[nown], 0\n"
      "s_add_i32 s94, s94, s95\n"
      "s_min_i32 s94, s94, s40\n"
      "s_max_i32 s74, s74, s94\n"
      "v_min3_u32 v9, v224, v228, v232\n"
      "v_min3_u32 v9, v9, v236, v240\n"
      "v_min3_u32 v9, v9, v244, v248\n"
      "v_min_u32 v9, v9, v252\n"
      "v_cmp_eq_u32 vcc, 0, v9\n"
      "s_nop 4\n"
      "s_cbranch_vccz .Lmidok2_%=\n"
      "s_add_u32 s84, s84, 1\n"
      "s_and_b32 s94, s84, 63\n"
      "s_cbranch_scc1 .Lmidre2_%=\n"
      "v_mov_b32 v9, 8192\n"
      "global_load_dword v9, v9, s[98:99] sc1\n"
      "s_memrealtime s[82:83]\n"
      "s_waitcnt vmcnt(0) lgkmcnt(0)\n"
      "v_readfirstlane_b32 s94, v9\n"
      "s_sub_u32 s95, s82, s80\n"
      "s_cmp_eq_u32 s94, s75\n"
      "s_cbranch_scc1 .Lmiddead2_%=\n"
      "s_cmp_lt_u32 s95, 30000000\n"
      "s_cbranch_scc1 .Lmidre2_%=\n"
      "s_branch .Lmiddead2_%=\n"
      ".Lmiddead2_%=:\n"
      "v_mov_b32 v9, 8192\n"
      "v_mov_b32 v8, s75\n"
      "global_store_dword v9, v8, s[98:99] sc1\n"
      "s_mov_b32 s79, 0x7fc00000\n"
      "s_mov_b32 s74, %[nb]\n"
      ".Lmidok2_%=:\n"
      "s_mov_b64 s[64:65], s[86:87]\n"
      "s_mov_b32 s66, s88\n"
      "s_branch .Lmidret2_%=\n"
      ".Lflush0_%=:\n"
      "s_lshl_b32 s85, s42, 17\n"
      "s_nop 11\n"
      "ds_write_b128 v3, v[20:23] offset:0\n"
      "ds_write_b128 v3, v[24:27] offset:4096\n"
      "ds_write_b128 v3, v[28:31] offset:8192\n"
      "s_waitcnt lgkmcnt(0)\n"
      "s_barrier\n"
      "ds_read_b128 v[20:23], v5 offset:0\n"
      "ds_read_b128 v[24:27], v6 offset:4096\n"
      "ds_read_b128 v[28:31], v7 offset:8192\n"
      "s_waitcnt lgkmcnt(0)\n"
      "v_add_f32 v16, v16, v20\n"
      "v_add_f32 v17, v17, v21\n"
      "v_add_f32 v18, v18, v22\n"
      "v_add_f32 v19, v19, v23\n"
      "v_add_f32 v24, v24, v28\n"
      "v_add_f32 v25, v25, v29\n"
      "v_add_f32 v26, v26, v30\n"
      "v_add_f32 v27, v27, v31\n"
      "v_add_f32 v16, v16, v24\n"
      "v_add_f32 v17, v17, v25\n"
      "v_add_f32 v18, v18, v26\n"
      "v_add_f32 v19, v19, v27\n"
      "v_mul_f32 v16, s79, v16\n"
      "v_mul_f32 v17, s79, v17\n"
      "v_mul_f32 v18, s79, v18\n"
      "v_mul_f32 v19, s79, v19\n"
      "v_cmp_gt_f32 s[86:87], 0, v16\n"
      "v_cmp_gt_f32 s[88:89], 0, v17\n"
      "v_cmp_gt_f32 s[90:91], 0, v18\n"
      "v_cmp_gt_f32 s[92:93], 0, v19\n"
      "v_cndmask_b32 v16, v16, 0, s[86:87]\n"
      "v_cndmask_b32 v17, v17, 0, s[88:89]\n"
      "v_cndmask_b32 v18, v18, 0, s[90:91]\n"
      "v_cndmask_b32 v19, v19, 0, s[92:93]\n"
      "buffer_store_dwordx4 v[16:19], v2, s[68:71], s85 offen nt\n"
      "s_branch .Lend_%=\n"
      ".Lflush1_%=:\n"
      "s_lshl_b32 s85, s42, 17\n"
      "s_nop 11\n"
      "ds_write_b128 v3, v[36:39] offset:12288\n"
      "ds_write_b128 v3, v[40:43] offset:16384\n"
      "ds_write_b128 v3, v[44:47] offset:20480\n"
      "s_waitcnt lgkmcnt(0)\n"
      "s_barrier\n"
      "ds_read_b128 v[36:39], v5 offset:12288\n"
      "ds_read_b128 v[40:43], v6 offset:16384\n"
      "ds_read_b128 v[44:47], v7 offset:20480\n"
      "s_waitcnt lgkmcnt(0)\n"
      "v_add_f32 v32, v32, v36\n"
      "v_add_f32 v33, v33, v37\n"
      "v_add_f32 v34, v34, v38\n"
      "v_add_f32 v35, v35, v39\n"
      "v_add_f32 v40, v40, v44\n"
      "v_add_f32 v41, v41, v45\n"
      "v_add_f32 v42, v42, v46\n"
      "v_add_f32 v43, v43, v47\n"
      "v_add_f32 v32, v32, v40\n"
      "v_add_f32 v33, v33, v41\n"
      "v_add_f32 v34, v34, v42\n"
      "v_add_f32 v35, v35, v43\n"
      "v_mul_f32 v32, s79, v32\n"
      "v_mul_f32 v33, s79, v33\n"
      "v_mul_f32 v34, s79, v34\n"
      "v_mul_f32 v35, s79, v35\n"
      "v_cmp_gt_f32 s[86:87], 0, v32\n"
      "v_cmp_gt_f32 s[88:89], 0, v33\n"
      "v_cmp_gt_f32 s[90:91], 0, v34\n"
      "v_cmp_gt_f32 s[92:93], 0, v35\n"
      "v_cndmask_b32 v32, v32, 0, s[86:87]\n"
      "v_cndmask_b32 v33, v33, 0, s[88:89]\n"
      "v_cndmask_b32 v34, v34, 0, s[90:91]\n"
      "v_cndmask_b32 v35, v35, 0, s[92:93]\n"
      "buffer_store_dwordx4 v[32:35], v2, s[68:71], s85 offen nt\n"
      "s_branch .Lend_%=\n"
      ".Lflush2_%=:\n"
      "s_lshl_b32 s85, s42, 17\n"
      "s_nop 11\n"
      "ds_write_b128 v3, v[52:55] offset:24576\n"
      "ds_write_b128 v3, v[56:59] offset:28672\n"
      "ds_write_b128 v3, v[60:63] offset:32768\n"
      "s_waitcnt lgkmcnt(0)\n"
      "s_barrier\n"
      "ds_read_b128 v[52:55], v5 offset:24576\n"
      "ds_read_b128 v[56:59], v6 offset:28672\n"
      "ds_read_b128 v[60:63], v7 offset:32768\n"
      "s_waitcnt lgkmcnt(0)\n"
      "v_add_f32 v48, v48, v52\n"
      "v_add_f32 v49, v49, v53\n"
      "v_add_f32 v50, v50, v54\n"
      "v_add_f32 v51, v51, v55\n"
      "v_add_f32 v56, v56, v60\n"
      "v_add_f32 v57, v57, v61\n"
      "v_add_f32 v58, v58, v62\n"
      "v_add_f32 v59, v59, v63\n"
      "v_add_f32 v48, v48, v56\n"
      "v_add_f32 v49, v49, v57\n"
      "v_add_f32 v50, v50, v58\n"
      "v_add_f32 v51, v51, v59\n"
      "v_mul_f32 v48, s79, v48\n"
      "v_mul_f32 v49, s79, v49\n"
      "v_mul_f32 v50, s79, v50\n"
      "v_mul_f32 v51, s79, v51\n"
      "v_cmp_gt_f32 s[86:87], 0, v48\n"
      "v_cmp_gt_f32 s[88:89], 0, v49\n"
      "v_cmp_gt_f32 s[90:91], 0, v50\n"
      "v_cmp_gt_f32 s[92:93], 0, v51\n"
      "v_cndmask_b32 v48, v48, 0, s[86:87]\n"
      "v_cndmask_b32 v49, v49, 0, s[88:89]\n"
      "v_cndmask_b32 v50, v50, 0, s[90:91]\n"
      "v_cndmask_b32 v51, v51, 0, s[92:93]\n"
      "buffer_store_dwordx4 v[48:51], v2, s[68:71], s85 offen nt\n"
      ".Lend_%=:\n"
      "s_nop 1\n"
      :
      : [wid] "s"(wid), [ldsb] "s"(lds_base), [ocol] "s"(ocol), [waddr] "s"(waddr), [out] "s"(P.out),
        [fl] "s"(ws + PROG_OFF), [po] "s"((int)(src * 4 + wid) * 128), [cto] "s"((int)(CTR_OFF - PROG_OFF) + (b * 4 + wid) * 4),
        [rot] "s"(rot), [nb] "s"(nb), [sxp] "s"(P.sx), [swp] "s"(P.sw),
        [nown] "s"(n_own), [nownr] "s"(n_own + RFIRST), [ownu] "s"(P.rank * NRT + pi - 7 * RFIRST), [srcu] "s"((int)src * NRT - n_own),
        [opo] "s"((P.rank * 4 + wid) * 128)
      :
        "v0", "v1", "v2", "v3", "v4", "v5", "v6", "v7", "v8", "v9", "v10", "v11", "v12", "v13", "v16", "v17",
        "v18", "v19", "v20", "v21", "v22", "v23", "v24", "v25", "v26", "v27", "v28", "v29", "v30", "v31", "v32", "v33",
        "v34", "v35", "v36", "v37", "v38", "v39", "v40", "v41", "v42", "v43", "v44", "v45", "v46", "v47", "v48", "v49",
        "v50", "v51", "v52", "v53", "v54", "v55", "v56", "v57", "v58", "v59", "v60", "v61", "v62", "v63", "v64", "v65",
        "v66", "v67", "v68", "v69", "v70", "v71", "v72", "v73", "v74", "v75", "v76", "v77", "v78", "v79", "v80", "v81",
        "v82", "v83", "v84", "v85", "v86", "v87", "v88", "v89", "v90", "v91", "v92", "v93", "v94", "v95", "v96", "v97",
        "v98", "v99", "v100", "v101", "v102", "v103", "v104", "v105", "v106", "v107", "v108", "v109", "v110", "v111", "v112", "v113",
        "v114", "v115", "v116", "v117", "v118", "v119", "v120", "v121", "v122", "v123", "v124", "v125", "v126", "v127", "v128", "v129",
        "v130", "v131", "v132", "v133", "v134", "v135", "v136", "v137", "v138", "v139", "v140", "v141", "v142", "v143", "v144", "v145",
        "v146", "v147", "v148", "v149", "v150", "v151", "v152", "v153", "v154", "v155", "v156", "v157", "v158", "v159", "v160", "v161",
        "v162", "v163", "v164", "v165", "v166", "v167", "v168", "v169", "v170", "v171", "v172", "v173", "v174", "v175", "v176", "v177",
        "v178", "v179", "v180", "v181", "v182", "v183", "v184", "v185", "v186", "v187", "v188", "v189", "v190", "v191", "v192", "v193",
        "v194", "v195", "v196", "v197", "v198", "v199", "v200", "v201", "v202", "v203", "v204", "v205", "v206", "v207", "v208", "v209",
        "v210", "v211", "v212", "v213", "v214", "v215", "v216", "v217", "v218", "v219", "v220", "v221", "v222", "v223", "v224", "v225",
        "v226", "v227", "v228", "v229", "v230", "v231", "v232", "v233", "v234", "v235", "v236", "v237", "v238", "v239", "v240", "v241",
        "v242", "v243", "v244", "v245", "v246", "v247", "v248", "v249", "v250", "v251", "v252", "v253", "v254", "v255", "a0", "a1",
        "a2", "a3", "a4", "a5", "a6", "a7", "a8", "a9", "a10", "a11", "a12", "a13", "a14", "a15", "a16", "a17",
        "a18", "a19", "a20", "a21", "a22", "a23", "a24", "a25", "a26", "a27", "a28", "a29", "a30", "a31", "a32", "a33",
        "a34", "a35", "a36", "a37", "a38", "a39", "a40", "a41", "a42", "a43", "a44", "a45", "a46", "a47", "a48", "a49",
        "a50", "a51", "a52", "a53", "a54", "a55", "a56", "a57", "a58", "a59", "a60", "a61", "a62", "a63", "a64", "a65",
        "a66", "a67", "a68", "a69", "a70", "a71", "a72", "a73", "a74", "a75", "a76", "a77", "a78", "a79", "a80", "a81",
        "a82", "a83", "a84", "a85", "a86", "a87", "a88", "a89", "a90", "a91", "a92", "a93", "a94", "a95", "a96", "a97",
        "a98", "a99", "a100", "a101", "a102", "a103", "a104", "a105", "a106", "a107", "a108", "a109", "a110", "a111", "a112", "a113",
        "a114", "a115", "a116", "a117", "a118", "a119", "a120", "a121", "a122", "a123", "a124", "a125", "a126", "a127", "a128", "a129",
        "a130", "a131", "a132", "a133", "a134", "a135", "a136", "a137", "a138", "a139", "a140", "a141", "a142", "a143", "a144", "a145",
        "a146", "a147", "a148", "a149", "a150", "a151", "a152", "a153", "a154", "a155", "a156", "a157", "a158", "a159", "a160", "a161",
        "a162", "a163", "a164", "a165", "a166", "a167", "a168", "a169", "a170", "a171", "a172", "a173", "a174", "a175", "a176", "a177",
        "a178", "a179", "a180", "a181", "a182", "a183", "a184", "a185", "a186", "a187", "a188", "a189", "a190", "a191", "a192", "a193",
        "a194", "a195", "a196", "a197", "a198", "a199", "a200", "a201", "a202", "a203", "a204", "a205", "a206", "a207", "a208", "a209",
        "a210", "a211", "a212", "a213", "a214", "a215", "a216", "a217", "a218", "a219", "a220", "a221", "a222", "a223", "a224", "a225",
        "a226", "a227", "a228", "a229", "a230", "a231", "a232", "a233", "a234", "a235", "a236", "a237", "a238", "a239", "a240", "a241",
        "a242", "a243", "a244", "a245", "a246", "a247", "a248", "a249", "a250", "a251", "a252", "a253", "a254", "a255", "s36", "s37",
        "s38", "s39", "s40", "s41", "s42", "s43", "s44", "s45", "s46", "s47", "s48", "s49", "s50", "s51", "s52", "s53",
        "s54", "s55", "s56", "s57", "s58", "s59", "s60", "s61", "s62", "s63", "s64", "s65", "s66", "s67", "s68", "s69",
        "s70", "s71", "s72", "s73", "s74", "s75", "s76", "s77", "s78", "s79", "s80", "s81", "s82", "s83", "s84", "s85",
        "s86", "s87", "s88", "s89", "s90", "s91", "s92", "s93", "s94", "s95", "s96", "s97", "s98", "s99", "vcc", "scc",
        "memory"
  );
}

__global__ void __launch_bounds__(256, 1) gemm_ag_kernel(const Params P) {
  __shared__ __attribute__((aligned(16))) unsigned char lds[LDS_BYTES];
  const int b = blockIdx.x;
  const int wid = __builtin_amdgcn_readfirstlane(threadIdx.x >> 6);
  asm volatile("" ::"s"(P.x), "s"(P.ws), "s"(P.rank), "s"(P.peer[0]), "s"(P.peer[1]), "s"(P.peer[2]), "s"(P.peer[3]), "s"(P.peer[4]),
               "s"(P.peer[5]), "s"(P.peer[6]), "s"(P.w), "s"(P.sx), "s"(P.sw), "s"(P.out));
#define TSTART 0u
  unsigned* ctr = (unsigned*)(P.ws + CTR_OFF) + b * 4 + wid;
  if (b >= CONS0) {
    consumer(P, b, wid, ((unsigned)(uintptr_t)(LDS_AS unsigned char*)lds + 4095u) & ~4095u);
  } else if (b >= NPUSH) {
    helper(P, b - NPUSH, wid, ctr);
  } else {
    const int lane = __builtin_amdgcn_mbcnt_hi(~0u, __builtin_amdgcn_mbcnt_lo(~0u, 0u));
    const unsigned tstart = TSTART;
    const unsigned tk = __hip_atomic_load(ctr, __ATOMIC_RELAXED, __HIP_MEMORY_SCOPE_AGENT);
    pusher(P, b, wid, lane, tk, ctr, (volatile LDS_AS unsigned*)lds, tstart);
  }
}

}

extern "C" void kernel_launch(int rank, int nranks, ncclComm_t comm,
                              hipStream_t stream,
                              void* const* d_in, const int* in_sizes, int n_in,
                              void* d_out, int out_size,
                              const int* in_dtypes, int out_dtype,
                              void* const* d_out_all,
                              void* const* d_ws_all, size_t ws_bytes) {
  (void)comm; (void)in_dtypes; (void)out_dtype; (void)d_out_all;
  if (nranks != NR || n_in < 4 || ws_bytes < WS_NEED || in_sizes[0] != MLOC * KDIM || in_sizes[1] != KDIM * NTOT ||
      out_size != NR * MLOC * NLOC || (unsigned)rank >= (unsigned)NR) {
    fprintf(stderr, "kernel_launch: unexpected problem shape\n");
    return;
  }
  Params P;
  P.x = (const unsigned char*)d_in[0];
  P.w = (const unsigned char*)d_in[1] + (size_t)rank * NLOC;
  P.sx = (const float*)d_in[2];
  P.sw = (const float*)d_in[3];
  P.out = (float*)d_out;
  P.ws = (unsigned char*)d_ws_all[rank];
  for (int p = 0; p < NR - 1; ++p) P.peer[p] = (unsigned char*)d_ws_all[(rank + 1 + p) % NR];
  P.pad = 0;
  P.rank = rank;
  hipLaunchKernelGGL(gemm_ag_kernel, dim3(GRID), dim3(256), 0, stream, P);
  HIP_CHECK(hipGetLastError());
}
```
